```python
import numpy as np

kernel_name = "gpu_gemm_ar_m4096_k4096_n4096_e4m3e4m3_dqrelu"


ABSMAX_RTOL = 0.03

COMPUTE_DTYPE = "float32"
EPILOGUE = "dequant_relu"
IN_QUANT = None
OUT_QUANT = None
IN_FP8 = "e4m3e4m3"
IN_DTYPES = {"x": "e4m3", "w_mat": "e4m3",
             "scale_x": "float32", "scale_w": "float32"}
WORLD_SIZE = 8

def _snap_fp8(x, mant_bits, max_abs, min_normal):
    a = np.abs(x.astype(np.float64))
    _m, _e = np.frexp(a)
    q = np.where(a >= min_normal, np.ldexp(1.0, _e - 1 - mant_bits),
                 min_normal * 2.0 ** -mant_bits)
    snapped = np.minimum(np.round(a / q) * q, max_abs)
    return (np.sign(x) * snapped).astype(np.float32)

def setup_inputs():

    x = _snap_fp8(np.random.randn(4096, 4096).astype(np.float32), 3, 448.0, 2.0 ** -6)
    w_mat = _snap_fp8(np.random.randn(4096, 4096).astype(np.float32), 3, 448.0, 2.0 ** -6)
    scale_x = (0.02 * (1.0 + np.random.rand(1))).astype(np.float32)
    scale_w = (0.02 * (1.0 + np.random.rand(1))).astype(np.float32)
    return {"x": x, "w_mat": w_mat, "scale_x": scale_x, "scale_w": scale_w}

def reference(x, w_mat, scale_x, scale_w):
    acc = (x @ w_mat).astype(np.float32)
    y = (acc * (scale_x * scale_w)).astype(np.float32)
    return np.maximum(y, 0.0).astype(np.float32)
```

```cpp
#include <hip/hip_runtime.h>
#include <rccl/rccl.h>
#include <cstdio>
#include <cstdint>

#define HIP_CHECK(expr) do { hipError_t _e = (expr); if (_e != hipSuccess) { \
    fprintf(stderr, "HIP error %s at %s:%d\n", hipGetErrorString(_e), __FILE__, __LINE__); \
    return; } } while (0)

#define LAS __attribute__((address_space(3)))
typedef int i32x8 __attribute__((ext_vector_type(8)));
typedef int i32x4 __attribute__((ext_vector_type(4)));
typedef unsigned u32x4 __attribute__((ext_vector_type(4)));
typedef unsigned u32x2 __attribute__((ext_vector_type(2)));
typedef float f32x4 __attribute__((ext_vector_type(4)));
typedef unsigned short u16x2 __attribute__((ext_vector_type(2)));

namespace {
constexpr int NRANK = 8;
constexpr int NCOMM = 16, NGEMM = 240, NWG = NCOMM + NGEMM;
constexpr unsigned OPER_BYTES = 16u << 20;
constexpr unsigned BUF_BYTES = 2u * OPER_BYTES;
constexpr unsigned FLAG_OFF = 2u * BUF_BYTES;
constexpr unsigned FLAG_BYTES = 32u * 32u * 4u * 2u;
constexpr unsigned CTR_OFF = FLAG_OFF + (64u << 10);
constexpr unsigned TMO_OFF = FLAG_OFF + (128u << 10);
constexpr unsigned WS_USED = FLAG_OFF + (192u << 10);
constexpr unsigned SHARD_BYTES = 2u << 20;
constexpr unsigned long long DEADLINE_TICKS = 50000000ull;

constexpr unsigned L_A0 = 0, L_A0B = 20480;
constexpr unsigned L_A1 = 40960, L_A1B = 16384;
constexpr unsigned L_B = 73728, L_BB = 32768;
constexpr unsigned L_CTRL = 139264;
constexpr unsigned LDS_BYTES = L_CTRL + 64;

struct Params {
  const unsigned char* x; const unsigned char* w; const float* sx; const float* sw; float* out;
  char* ws[NRANK]; int rank;
};

__device__ __forceinline__ __amdgpu_buffer_rsrc_t mk_rsrc(const void* p, unsigned bytes) {
  return __builtin_amdgcn_make_buffer_rsrc((void*)p, 0, bytes, 0x00020000);
}
__device__ __forceinline__ unsigned ld_sys(const unsigned* p) {
  return __hip_atomic_load(p, __ATOMIC_RELAXED, __HIP_MEMORY_SCOPE_SYSTEM);
}
#define CFENCE asm volatile("" ::: "memory")
#define WAIT_V(n) asm volatile("s_waitcnt vmcnt(" #n ")" ::: "memory")
#define WAIT_L(n) asm volatile("s_waitcnt lgkmcnt(" #n ")" ::: "memory")
#define BAR do { CFENCE; __builtin_amdgcn_s_barrier(); CFENCE; } while (0)
#define SCHED __builtin_amdgcn_sched_barrier(0)

__device__ __forceinline__ void comm_role(const Params& p, LAS unsigned char* lds, unsigned call, int wid, int lane) {
  const int c = blockIdx.x, o = c >> 3, g2 = (c >> 2) & 1, q4 = c & 3;
  const int rank = p.rank & 7;
  LAS unsigned* cnt = (LAS unsigned*)lds;
  LAS unsigned char* tsc = lds + 1024 + wid * 1024;
  if (threadIdx.x < 32) cnt[threadIdx.x] = 0;
  const int ub = 4 * q4 + (wid >> 1), h = wid & 1;
  const __amdgpu_buffer_rsrc_t srcR = mk_rsrc(o ? p.w : p.x, SHARD_BYTES);
  const unsigned lpart = o ? (unsigned)lane * 4096u : (unsigned)((lane & 15) * 512 + (lane >> 4) * 16);
  u32x4 d[32];
#pragma unroll
  for (int n = 0; n < 32; ++n) {
    const int ob = 2 * n + g2, s = ob >> 4, eb = ((ob & 15) + 2 * rank) & 15, u = eb * 16 + ub;
    const unsigned upart = o ? (unsigned)((s * 128 + h * 64) * 4096 + u * 16) : (unsigned)(u * 16 * 512 + s * 128 + h * 64);
    d[n] = __builtin_amdgcn_raw_buffer_load_b128(srcR, lpart, upart, 0);
  }
  __amdgpu_buffer_rsrc_t dstR[8];
  {
    const int rot = (c * 8 + wid) % 7;
#pragma unroll
    for (int q = 0; q < 7; ++q) { const int pr = (rank + 1 + (q + rot) % 7) & 7; dstR[q] = mk_rsrc(p.ws[pr], FLAG_OFF); }
    dstR[7] = mk_rsrc(p.ws[rank], FLAG_OFF);
  }
  char* flp = p.ws[0];
#pragma unroll
  for (int q = 1; q < 8; ++q) flp = ((lane & 7) == q) ? p.ws[q] : flp;
  flp += FLAG_OFF;
  const unsigned bufoff = (call & 1u) * BUF_BYTES + (unsigned)o * OPER_BYTES;
  const unsigned voff = (unsigned)lane * 16u;
  const unsigned tr_addr = (unsigned)(size_t)tsc + (unsigned)((lane >> 4) * 256 + (lane & 15) * 8);
  WAIT_L(0); BAR;

  auto signal = [&](int n) {
    unsigned old = 0;
    if (lane == 0) old = __hip_atomic_fetch_add(cnt + n, 1u, __ATOMIC_RELAXED, __HIP_MEMORY_SCOPE_WORKGROUP);
    old = __builtin_amdgcn_readfirstlane(old);
    if (old == 7u) {
      const int ob = 2 * n + g2, s = ob >> 4, eb = ((ob & 15) + 2 * rank) & 15;
      const unsigned idx = (unsigned)((((o * 16 + eb) * 32) + rank * 4 + s) * 4 + q4);
      if (lane < 8) __hip_atomic_store((unsigned short*)flp + idx, (unsigned short)call, __ATOMIC_RELAXED, __HIP_MEMORY_SCOPE_SYSTEM);
    }
  };
#pragma unroll
  for (int n = 0; n < 32; ++n) {
    const int ob = 2 * n + g2, s = ob >> 4, eb = ((ob & 15) + 2 * rank) & 15, u = eb * 16 + ub, ks = rank * 4 + s;
    u32x4 v = d[n];
    if (o) {
      *(LAS u32x4*)(tsc + lane * 16) = v;
      u32x2 lo, hi;
      asm volatile("s_waitcnt lgkmcnt(0)\n\tds_read_b64_tr_b8 %0, %2\n\tds_read_b64_tr_b8 %1, %2 offset:128\n\ts_waitcnt lgkmcnt(0)"
                   : "=&v"(lo), "=&v"(hi) : "v"(tr_addr) : "memory");
      v = (u32x4){lo.x, lo.y, hi.x, hi.y};
    }
    const unsigned doff = bufoff + (unsigned)(((ks * 256 + u) * 2 + h) * 1024);
#pragma unroll
    for (int q = 0; q < 8; ++q) __builtin_amdgcn_raw_buffer_store_b128(v, dstR[q], voff, doff, 17);
    if (n >= 1) { WAIT_V(8); signal(n - 1); }
  }
  WAIT_V(0); signal(31);
}

__device__ __forceinline__ i32x8 ld_frag(LAS unsigned char* a) {
  const i32x4 lo = *(const LAS i32x4*)a, hi = *(const LAS i32x4*)(a + 1024);
  return __builtin_shufflevector(lo, hi, 0, 1, 2, 3, 4, 5, 6, 7);
}

__device__ __forceinline__ void gemm_role(const Params& p, LAS unsigned char* lds, unsigned call, int wid, int lane) {
  const int g = blockIdx.x - NCOMM, idx = g >> 3;
  const int tj = 2 * (g & 7) + (idx & 1), tr = idx >> 1;
  const int wr = wid >> 2, wc = wid & 3;
  const int rank = p.rank & 7;
  char* myws = p.ws[rank];
  const int U = 17 * tr, UX = (tr == 14) ? 255 : U + 8;
  const __amdgpu_buffer_rsrc_t opR = mk_rsrc(myws + (size_t)(call & 1u) * BUF_BYTES, BUF_BYTES);
  const unsigned voff = (unsigned)lane * 16u;
  volatile LAS unsigned* ctrl = (volatile LAS unsigned*)(lds + L_CTRL);
  unsigned* tmo = (unsigned*)(myws + TMO_OFF);

  const unsigned sA0a = (unsigned)(U * 2 + wid) * 1024u;
  const unsigned sA0x = (wid < 6) ? (unsigned)(U * 2 + 12 + wid) * 1024u : (unsigned)(UX * 2 + wid - 6) * 1024u;
  const unsigned sA1 = (unsigned)((U + 9) * 2 + wid) * 1024u;
  const unsigned sB = OPER_BYTES + (unsigned)(tj * 32 + wid) * 1024u;
  const unsigned lw = (unsigned)wid * 1024u;
#define GLDS(ldsoff, soff) __builtin_amdgcn_raw_ptr_buffer_load_lds(opR, (LAS void*)(lds + (ldsoff)), 16, voff, (soff), 0, 17)
#define STAGE_A0(b, kb) do { GLDS(L_A0 + (b) * L_A0B + lw, (kb) + sA0a); GLDS(L_A0 + (b) * L_A0B + lw + 8192, (kb) + sA0a + 8192); \
                             if (wr) GLDS(L_A0 + (b) * L_A0B + lw + 12288, (kb) + sA0x); } while (0)
#define STAGE_A1(b, kb) do { GLDS(L_A1 + (b) * L_A1B + lw, (kb) + sA1); GLDS(L_A1 + (b) * L_A1B + lw + 8192, (kb) + sA1 + 8192); } while (0)
#define STAGE_B(b, bj, kb) do { GLDS(L_B + (b) * L_BB + (bj) * 16384 + lw, (kb) + sB + (bj) * 16384); \
                                GLDS(L_B + (b) * L_BB + (bj) * 16384 + lw + 8192, (kb) + sB + (bj) * 16384 + 8192); } while (0)
#define WAIT_TILE do { if (wr) WAIT_V(7); else WAIT_V(6); } while (0)

  LAS unsigned char* rA0 = lds + L_A0 + wr * 10240 + lane * 16;
  LAS unsigned char* rA1 = lds + L_A1 + wr * 8192 + lane * 16;
  LAS unsigned char* rB = lds + L_B + wc * 4096 + lane * 16;
#define LDA0(b) do { _Pragma("unroll") for (int m = 0; m < 5; ++m) At[m] = ld_frag(rA0 + (b) * L_A0B + m * 2048); } while (0)
#define LDA1(b) do { _Pragma("unroll") for (int m = 0; m < 4; ++m) At[m] = ld_frag(rA1 + (b) * L_A1B + m * 2048); } while (0)
#define LDB(dst, b, bj) do { _Pragma("unroll") for (int n = 0; n < 2; ++n) dst[n] = ld_frag(rB + (b) * L_BB + (bj) * 16384 + n * 2048); } while (0)
#define MMA0(bj, Bt) do { __builtin_amdgcn_s_setprio(1); _Pragma("unroll") for (int m = 0; m < 5; ++m) _Pragma("unroll") for (int n = 0; n < 2; ++n) \
    acc0[bj][m][n] = __builtin_amdgcn_mfma_scale_f32_16x16x128_f8f6f4(Bt[n], At[m], acc0[bj][m][n], 0, 0, 0, 0, 0, 0); __builtin_amdgcn_s_setprio(0); } while (0)
#define MMA1(bj, Bt) do { __builtin_amdgcn_s_setprio(1); _Pragma("unroll") for (int m = 0; m < 4; ++m) _Pragma("unroll") for (int n = 0; n < 2; ++n) \
    acc1[bj][m][n] = __builtin_amdgcn_mfma_scale_f32_16x16x128_f8f6f4(Bt[n], At[m], acc1[bj][m][n], 0, 0, 0, 0, 0, 0); __builtin_amdgcn_s_setprio(0); } while (0)

  f32x4 acc0[2][5][2], acc1[2][4][2];
#pragma unroll
  for (int b = 0; b < 2; ++b)
#pragma unroll
    for (int n = 0; n < 2; ++n) {
#pragma unroll
      for (int m = 0; m < 5; ++m) acc0[b][m][n] = (f32x4){0.f, 0.f, 0.f, 0.f};
#pragma unroll
      for (int m = 0; m < 4; ++m) acc1[b][m][n] = (f32x4){0.f, 0.f, 0.f, 0.f};
    }
  i32x8 At[5], B0[2], B1[2];

  const char* fbase = myws + FLAG_OFF;
  const char* fpa = fbase + (size_t)(((lane < 32 ? tr : 16 + tj) * 32 + (lane & 31)) * 8);
  const char* fpb = fbase + (size_t)(((lane < 32 ? tr + 1 : 16 + tj) * 32 + (lane & 31)) * 8);
  u32x2 pa = {0u, 0u}, pb = {0u, 0u};
  unsigned done = 0, bad = 0;
  const unsigned short c16 = (unsigned short)call;
  const u16x2 cc = {c16, c16};
  auto issue_poll = [&]() {
    asm volatile("global_load_dwordx2 %0, %2, off sc0 sc1\n\tglobal_load_dwordx2 %1, %3, off sc0 sc1"
                 : "=&v"(pa), "=&v"(pb) : "v"(fpa), "v"(fpb) : "memory");
  };
  auto pksub = [&](unsigned v) -> unsigned { u16x2 a = __builtin_bit_cast(u16x2, v); a -= cc; return __builtin_bit_cast(unsigned, a); };
  auto ready_mask = [&]() -> unsigned {
    const unsigned w0 = pa[0], w1 = pa[1], w2 = pb[0], w3 = pb[1];
    const unsigned dd = (pksub(w0) | pksub(w1)) | (pksub(w2) | pksub(w3));
    const unsigned long long bl = __builtin_amdgcn_ballot_w64((dd & 0xFFFEFFFEu) == 0u);
    return (unsigned)__builtin_amdgcn_readfirstlane((unsigned)bl & (unsigned)(bl >> 32) & ~done);
  };
  auto decide = [&](int slot) {
    asm volatile("" : "+v"(pa), "+v"(pb));
    unsigned m = ready_mask();
    if (m == 0u) {
      const unsigned long long t0 = __builtin_amdgcn_s_memrealtime();
      while (m == 0u) {
        if (bad == 0u) {
          issue_poll();
          asm volatile("s_waitcnt vmcnt(0)" : "+v"(pa), "+v"(pb) :: "memory");
          m = ready_mask();
          if (m == 0u) {
            const unsigned tv = (unsigned)__builtin_amdgcn_readfirstlane(__hip_atomic_load(tmo, __ATOMIC_RELAXED, __HIP_MEMORY_SCOPE_AGENT));
            if (tv == call || __builtin_amdgcn_s_memrealtime() - t0 > DEADLINE_TICKS) {
              if (lane == 0) { __hip_atomic_store(tmo, call, __ATOMIC_RELAXED, __HIP_MEMORY_SCOPE_AGENT); ctrl[2] = 1u; }
              bad = 1u;
            }
            __builtin_amdgcn_s_sleep(1);
          }
        } else {
          m = ~done;
        }
        m = (unsigned)__builtin_amdgcn_readfirstlane(m);
        bad = (unsigned)__builtin_amdgcn_readfirstlane(bad);
      }
    }
    const unsigned ks = (unsigned)__builtin_ctz(m);
    done = (unsigned)__builtin_amdgcn_readfirstlane(done | (1u << ks));
    if (lane == 0) ctrl[slot] = ks;
    issue_poll();
  };
#define READ_D(slot) ((unsigned)__builtin_amdgcn_readfirstlane(ctrl[slot]) * (512u * 1024u))

  if (wid == 0) {
    if (lane == 0) ctrl[2] = 0u;
    issue_poll(); asm volatile("s_waitcnt vmcnt(0)" : "+v"(pa), "+v"(pb) :: "memory");
    decide(0);
    asm volatile("s_waitcnt vmcnt(0)" : "+v"(pa), "+v"(pb) :: "memory");
    decide(1);
  }
  WAIT_L(0); BAR;
  unsigned kb0 = READ_D(0), kb1 = READ_D(1);
  WAIT_L(0); BAR;
  STAGE_B(0, 0, kb0); STAGE_A0(0, kb0); STAGE_B(0, 1, kb0); STAGE_A1(0, kb0);
  if (wr == 1) BAR;
  WAIT_V(4); BAR;
  STAGE_B(1, 0, kb1); STAGE_A0(1, kb1); STAGE_B(1, 1, kb1);
  WAIT_TILE; BAR;

  for (int it = 0; it < 16; ++it) {
    const bool more = it < 15;
    unsigned kb2 = 0, kb3 = 0;
    if (wid == 0 && more) decide(0);
    LDB(B0, 0, 0); SCHED; LDA0(0); STAGE_A1(1, kb1);
    WAIT_L(10); BAR; WAIT_L(0); MMA0(0, B0); BAR; SCHED;
    if (more) kb2 = READ_D(0);
    LDB(B1, 0, 1); if (more) STAGE_B(0, 0, kb2);
    BAR; WAIT_L(0); MMA0(1, B1); BAR;
    LDA1(0); if (more) STAGE_A0(0, kb2);
    BAR; WAIT_L(0); MMA1(0, B0); BAR; SCHED;
    if (more) { STAGE_B(0, 1, kb2); WAIT_TILE; } else { WAIT_V(0); }
    BAR; MMA1(1, B1); BAR;
    if (wid == 0 && more) decide(1);
    LDB(B0, 1, 0); SCHED; LDA0(1); if (more) STAGE_A1(0, kb2);
    WAIT_L(10); BAR; WAIT_L(0); MMA0(0, B0); BAR; SCHED;
    if (more) kb3 = READ_D(1);
    LDB(B1, 1, 1); if (more) STAGE_B(1, 0, kb3);
    BAR; WAIT_L(0); MMA0(1, B1); BAR;
    LDA1(1); if (more) STAGE_A0(1, kb3);
    BAR; WAIT_L(0); MMA1(0, B0); BAR; SCHED;
    if (more) { STAGE_B(1, 1, kb3); WAIT_TILE; }
    BAR; MMA1(1, B1); BAR;
    kb1 = kb3;
  }
  if (wr == 0) BAR;

  const float scale = p.sx[0] * p.sw[0];
  const bool poison = (ctrl[2] != 0u) || (__hip_atomic_load(tmo, __ATOMIC_RELAXED, __HIP_MEMORY_SCOPE_AGENT) == call);
  const float qn = __builtin_nanf("");
  const int fr = lane & 15, fq = lane >> 4;
  float* obase = p.out + (size_t)fr * 4096 + tj * 256 + wc * 32 + fq * 4;
  auto put = [&](const f32x4& a, int unit, int bj, int n) {
    f32x4 y = a * scale;
#pragma unroll
    for (int e = 0; e < 4; ++e) y[e] = (y[e] < 0.f) ? 0.f : y[e];
    if (poison) y = (f32x4){qn, qn, qn, qn};
    *(f32x4*)(obase + (size_t)unit * 16 * 4096 + bj * 128 + n * 16) = y;
  };
#pragma unroll
  for (int m = 0; m < 5; ++m) {
    const int slot = wr * 5 + m;
    if (slot < 9 || tr == 14) {
      const int unit = (slot < 9) ? U + slot : 255;
#pragma unroll
      for (int bj = 0; bj < 2; ++bj)
#pragma unroll
        for (int n = 0; n < 2; ++n) put(acc0[bj][m][n], unit, bj, n);
    }
  }
#pragma unroll
  for (int m = 0; m < 4; ++m) {
    const int unit = U + 9 + wr * 4 + m;
#pragma unroll
    for (int bj = 0; bj < 2; ++bj)
#pragma unroll
      for (int n = 0; n < 2; ++n) put(acc1[bj][m][n], unit, bj, n);
  }
}
}

__global__ void __launch_bounds__(512, 2) fused_gemm_ag_kernel(Params p) {
  extern __shared__ __attribute__((aligned(16))) unsigned char smem_dyn[];
  LAS unsigned char* lds = (LAS unsigned char*)smem_dyn;
  const int wid = __builtin_amdgcn_readfirstlane(threadIdx.x >> 6), lane = threadIdx.x & 63;
  unsigned* ctr = (unsigned*)(p.ws[p.rank & 7] + CTR_OFF) + blockIdx.x * 16;
  const unsigned call = (unsigned)__builtin_amdgcn_readfirstlane(ld_sys(ctr)) + 1u;
  if (blockIdx.x < NCOMM) comm_role(p, lds, call, wid, lane);
  else gemm_role(p, lds, call, wid, lane);
  if (threadIdx.x == 0) __hip_atomic_store(ctr, call, __ATOMIC_RELAXED, __HIP_MEMORY_SCOPE_SYSTEM);
}

extern "C" void kernel_launch(int rank, int nranks, ncclComm_t comm,
                              hipStream_t stream,
                              void* const* d_in, const int* in_sizes, int n_in,
                              void* d_out, int out_size,
                              const int* in_dtypes, int out_dtype,
                              void* const* d_out_all,
                              void* const* d_ws_all, size_t ws_bytes) {
  (void)comm; (void)in_sizes; (void)n_in; (void)out_size; (void)in_dtypes; (void)out_dtype; (void)d_out_all;
  if (nranks != NRANK || (unsigned)rank >= (unsigned)NRANK || ws_bytes < (size_t)WS_USED) {
    fprintf(stderr, "unsupported configuration\n"); return;
  }
  HIP_CHECK(hipFuncSetAttribute((const void*)fused_gemm_ag_kernel, hipFuncAttributeMaxDynamicSharedMemorySize, (int)LDS_BYTES));
  Params p;
  p.x = (const unsigned char*)d_in[0]; p.w = (const unsigned char*)d_in[1];
  p.sx = (const float*)d_in[2]; p.sw = (const float*)d_in[3];
  p.out = (float*)d_out; p.rank = rank;
  for (int i = 0; i < NRANK; ++i) p.ws[i] = (char*)d_ws_all[i];
  hipLaunchKernelGGL(fused_gemm_ag_kernel, dim3(NWG), dim3(512), LDS_BYTES, stream, p);
  HIP_CHECK(hipGetLastError());
}
```

```cpp
#include <hip/hip_runtime.h>
#include <rccl/rccl.h>
#include <cstdio>
#include <cstdint>

#define HIP_CHECK(expr) do { hipError_t _e = (expr); if (_e != hipSuccess) { \
    fprintf(stderr, "HIP error %s at %s:%d\n", hipGetErrorString(_e), __FILE__, __LINE__); \
    return; } } while (0)

#define LAS __attribute__((address_space(3)))
typedef int i32x8 __attribute__((ext_vector_type(8)));
typedef int i32x4 __attribute__((ext_vector_type(4)));
typedef int i32x2 __attribute__((ext_vector_type(2)));
typedef unsigned u32x4 __attribute__((ext_vector_type(4)));
typedef unsigned u32x2 __attribute__((ext_vector_type(2)));
typedef float f32x4 __attribute__((ext_vector_type(4)));
typedef unsigned short u16x2 __attribute__((ext_vector_type(2)));

namespace {
constexpr int NRANK = 8;
constexpr int NCOMM = 16, NGEMM = 240, NWG = NCOMM + NGEMM;
constexpr unsigned OPER_BYTES = 16u << 20;
constexpr unsigned BUF_BYTES = 2u * OPER_BYTES;
constexpr unsigned FLAG_OFF = 2u * BUF_BYTES;
constexpr unsigned FLAG_BYTES = 32u * 32u * 4u * 2u;
constexpr unsigned CTR_OFF = FLAG_OFF + (64u << 10);
constexpr unsigned TMO_OFF = FLAG_OFF + (128u << 10);
constexpr unsigned PROG_OFF = FLAG_OFF + (160u << 10);
constexpr unsigned MBOX_OFF = FLAG_OFF + (192u << 10);
constexpr unsigned WS_USED = FLAG_OFF + (256u << 10);
constexpr unsigned SHARD_BYTES = 2u << 20;
#define PACE_D 6
#define RELAY_SLEEP 1
#define POLL_SLEEP 2
constexpr unsigned long long DEADLINE_TICKS = 50000000ull;

constexpr unsigned L_A0 = 0, L_A0B = 20480;
constexpr unsigned L_A1 = 40960, L_A1B = 16384;
constexpr unsigned L_B = 73728, L_BB = 32768;
constexpr unsigned L_CTRL = 139264;
constexpr unsigned C_RING = 1024;
constexpr unsigned LDS_BYTES = L_CTRL + 64;

struct Params {
  const unsigned char* x; const unsigned char* w; const float* sx; const float* sw; float* out;
  char* ws[NRANK]; int rank;
};
#define STAMP(i) do { } while (0)

__device__ __forceinline__ __amdgpu_buffer_rsrc_t mk_rsrc(const void* p, unsigned bytes) {
  return __builtin_amdgcn_make_buffer_rsrc((void*)p, 0, bytes, 0x00020000);
}
__device__ __forceinline__ unsigned ld_sys(const unsigned* p) {
  return __hip_atomic_load(p, __ATOMIC_RELAXED, __HIP_MEMORY_SCOPE_SYSTEM);
}
#define CFENCE asm volatile("" ::: "memory")
#define WAIT_V(n) asm volatile("s_waitcnt vmcnt(" #n ")" ::: "memory")
#define WAIT_L(n) asm volatile("s_waitcnt lgkmcnt(" #n ")" ::: "memory")
#define BAR do { CFENCE; __builtin_amdgcn_s_barrier(); CFENCE; } while (0)
#define SCHED __builtin_amdgcn_sched_barrier(0)

__device__ __forceinline__ void comm_role(const Params& p, LAS unsigned char* lds, const unsigned* ctr, unsigned& call_out, int wid, int lane) {
  const int c = blockIdx.x, g4 = c >> 2, q4 = c & 3;
  const int rank = p.rank & 7;
  LAS unsigned* cnt = (LAS unsigned*)lds;
  volatile LAS unsigned* allow = (volatile LAS unsigned*)(lds + 128);
  if (threadIdx.x < 32) cnt[threadIdx.x] = 0;
  if (threadIdx.x == 32) allow[0] = PACE_D;
  if (wid >= 4) {
    const unsigned call = (unsigned)__builtin_amdgcn_readfirstlane(ld_sys(ctr)) + 1u;
    call_out = call;
    WAIT_L(0); BAR;
    if (wid == 7) return;
    if (wid == 6) {
      const unsigned* pg = (const unsigned*)(p.ws[rank] + PROG_OFF) + (lane & 15);
      const unsigned base = call * 64u;
      const unsigned long long t0 = __builtin_amdgcn_s_memrealtime();
      unsigned cur = PACE_D;
      while (cur < 32u) {
        const unsigned v = ld_sys(pg);
        unsigned dn = (v - base) <= 32u ? v - base : 0u;
#pragma unroll
        for (int sh = 1; sh < 16; sh <<= 1) dn = min(dn, (unsigned)__shfl_xor((int)dn, sh));
        const unsigned na = (unsigned)__builtin_amdgcn_readfirstlane(dn) + PACE_D;
        if (na > cur) { cur = na; if (lane == 0) allow[0] = cur; }
        if (__builtin_amdgcn_s_memrealtime() - t0 > DEADLINE_TICKS) { cur = 64u; if (lane == 0) allow[0] = cur; }
        __builtin_amdgcn_s_sleep(1);
      }
      return;
    }
    const int e = 2 * c + (wid - 4);
    char* myws = p.ws[rank];
    const char* fp = myws + FLAG_OFF + (size_t)((e * 32 + (lane & 31)) * 8);
    int ctr_r, ctj, slot; bool tgt;
    if (e < 16) { ctr_r = (lane < 16) ? e : e - 1; ctj = lane & 15; slot = (lane < 16) ? 0 : 1; tgt = lane < 32 && (unsigned)ctr_r < 15u; }
    else { ctr_r = lane; ctj = e - 16; slot = 2; tgt = lane < 15; }
    unsigned long long* mb = (unsigned long long*)(myws + MBOX_OFF + (size_t)(((tgt ? ctr_r : 0) * 16 + ctj) * 128 + slot * 8));
    unsigned* tmo = (unsigned*)(myws + TMO_OFF);
    const unsigned short c16 = (unsigned short)call; const u16x2 cc = {c16, c16};
    unsigned last = 0;
    const unsigned long long t0 = __builtin_amdgcn_s_memrealtime();
    for (;;) {
      const unsigned long long fv = __hip_atomic_load((const unsigned long long*)fp, __ATOMIC_RELAXED, __HIP_MEMORY_SCOPE_SYSTEM);
      u16x2 a = __builtin_bit_cast(u16x2, (unsigned)fv), b = __builtin_bit_cast(u16x2, (unsigned)(fv >> 32));
      a -= cc; b -= cc;
      const unsigned dd = __builtin_bit_cast(unsigned, a) | __builtin_bit_cast(unsigned, b);
      const unsigned m = (unsigned)__builtin_amdgcn_ballot_w64((dd & 0xFFFEFFFEu) == 0u || ((lane & 31) >> 2) == rank);
      if (m != last) {
        if (tgt) __hip_atomic_store(mb, ((unsigned long long)call << 32) | m, __ATOMIC_RELAXED, __HIP_MEMORY_SCOPE_SYSTEM);
        last = m;
        if (m == 0xFFFFFFFFu) break;
      }
      if (__builtin_amdgcn_s_memrealtime() - t0 > DEADLINE_TICKS) {
        if (lane == 0) __hip_atomic_store(tmo, call, __ATOMIC_RELAXED, __HIP_MEMORY_SCOPE_AGENT);
        break;
      }
      __builtin_amdgcn_s_sleep(RELAY_SLEEP);
    }
    return;
  }
  LAS unsigned char* ring = lds + C_RING + wid * 8192;
  const __amdgpu_buffer_rsrc_t xR = mk_rsrc(p.x, SHARD_BYTES), wR = mk_rsrc(p.w, SHARD_BYTES);
  const unsigned lp_w = (unsigned)((lane >> 2) * 4096 + (lane & 3) * 16), lp_x = (unsigned)((lane & 15) * 512 + (lane >> 4) * 16);
  auto blk = [&](int n, int& o, int& eb, int& s) {
    const int idx = 4 * n + g4; int e;
    if (idx < 16) { o = 1; e = idx; s = 0; }
    else if (idx < 112) { const int j = idx - 16; e = (j & 31) >> 1; o = j & 1; s = (j >> 5) + o; }
    else { o = 0; e = idx - 112; s = 3; }
    eb = (e + 2 * rank) & 15;
  };
  auto fetch = [&](int n) {
    int o, eb, s; blk(n < 32 ? n : 31, o, eb, s);
    const int u0 = eb * 16 + 4 * q4;
    LAS unsigned char* dst = ring + (n & 3) * 2048;
    if (o) {
      const unsigned so = (unsigned)((s * 128 + 32 * wid) * 4096 + u0 * 16);
      __builtin_amdgcn_raw_ptr_buffer_load_lds(wR, (LAS void*)dst, 16, lp_w, so, 0, 0);
      __builtin_amdgcn_raw_ptr_buffer_load_lds(wR, (LAS void*)(dst + 1024), 16, lp_w, so + 16u * 4096u, 0, 0);
    } else {
      const unsigned so = (unsigned)((u0 + wid) * 8192 + s * 128);
      __builtin_amdgcn_raw_ptr_buffer_load_lds(xR, (LAS void*)dst, 16, lp_x, so, 0, 0);
      __builtin_amdgcn_raw_ptr_buffer_load_lds(xR, (LAS void*)(dst + 1024), 16, lp_x, so + 64u, 0, 0);
    }
  };
  const unsigned craw = ld_sys(ctr);
  fetch(0); fetch(1); fetch(2);
  const unsigned call = (unsigned)__builtin_amdgcn_readfirstlane(craw) + 1u;
  asm volatile("" :: "s"(call));
  call_out = call;
  __amdgpu_buffer_rsrc_t dstR[7];
  {
    const int rot = (c * 4 + wid) % 7;
#pragma unroll
    for (int q = 0; q < 7; ++q) { const int pr = (rank + 1 + (q + rot) % 7) & 7; dstR[q] = mk_rsrc(p.ws[pr], FLAG_OFF); }
  }
  char* flp = p.ws[0];
#pragma unroll
  for (int q = 1; q < 8; ++q) flp = ((lane & 7) == q) ? p.ws[q] : flp;
  flp += FLAG_OFF;
  asm volatile("" :: "v"(flp));
  const unsigned vo_w = (unsigned)((lane >> 5) * 2048 + ((lane >> 4) & 1) * 256 + (lane & 15) * 16), vo_x = (unsigned)lane * 16u;
  const unsigned dcommon = (call & 1u) * BUF_BYTES + (unsigned)(4 * q4) * 2048u;
  const unsigned db_w = dcommon + OPER_BYTES + (unsigned)((wid >> 1) * 1024 + (wid & 1) * 512), db_x = dcommon + (unsigned)wid * 2048u;
  unsigned* prog = (unsigned*)(p.ws[rank] + PROG_OFF) + c;
  const unsigned rd_w = (unsigned)(size_t)ring + (unsigned)((((lane >> 4) & 1) * 16 + ((lane & 15) >> 1)) * 64 + (lane >> 5) * 16 + (lane & 1) * 8);
  const unsigned rd_x = (unsigned)(size_t)ring + (unsigned)lane * 16u;
  WAIT_L(0); BAR;

  auto signal = [&](int n) {
    unsigned old = 0;
    if (lane == 0) old = __hip_atomic_fetch_add(cnt + n, 1u, __ATOMIC_RELAXED, __HIP_MEMORY_SCOPE_WORKGROUP);
    old = __builtin_amdgcn_readfirstlane(old);
    if (old == 3u) {
      int o, eb, s; blk(n, o, eb, s);
      const unsigned idx = (unsigned)((((o * 16 + eb) * 32) + rank * 4 + s) * 4 + q4);
      if (lane < 8 && lane != rank) __hip_atomic_store((unsigned short*)flp + idx, (unsigned short)call, __ATOMIC_RELAXED, __HIP_MEMORY_SCOPE_SYSTEM);
      if (lane == 0) __hip_atomic_fetch_max(prog, call * 64u + (unsigned)(n + 1), __ATOMIC_RELAXED, __HIP_MEMORY_SCOPE_SYSTEM);
    }
  };
#pragma unroll 1
  for (int n = 0; n < 32; ++n) {
    if (n == 0) WAIT_V(4); else if (n == 1) WAIT_V(18);
    int o, eb, s; blk(n, o, eb, s);
    u32x4 v0, v1;
    if (o) {
      const unsigned a = rd_w + (unsigned)((n & 3) * 2048);
      u32x2 a0, a1, b0, b1;
      asm volatile("ds_read_b64_tr_b8 %0, %4\n\tds_read_b64_tr_b8 %1, %4 offset:512\n\t"
                   "ds_read_b64_tr_b8 %2, %4 offset:32\n\tds_read_b64_tr_b8 %3, %4 offset:544\n\ts_waitcnt lgkmcnt(0)"
                   : "=&v"(a0), "=&v"(a1), "=&v"(b0), "=&v"(b1) : "v"(a) : "memory");
      v0 = (u32x4){a0.x, a0.y, a1.x, a1.y}; v1 = (u32x4){b0.x, b0.y, b1.x, b1.y};
    } else {
      const unsigned a = rd_x + (unsigned)((n & 3) * 2048);
      asm volatile("ds_read_b128 %0, %2\n\tds_read_b128 %1, %2 offset:1024\n\ts_waitcnt lgkmcnt(0)"
                   : "=&v"(v0), "=&v"(v1) : "v"(a) : "memory");
    }
    const unsigned voff = o ? vo_w : vo_x;
    const unsigned doff = (o ? db_w : db_x) + (unsigned)(((rank * 4 + s) * 256 + eb * 16) * 2048);
    const unsigned doff1 = doff + (o ? 4096u : 1024u);
    while ((unsigned)__builtin_amdgcn_readfirstlane(allow[0]) <= (unsigned)n) __builtin_amdgcn_s_sleep(1);
#pragma unroll
    for (int q = 0; q < 7; ++q) {
      __builtin_amdgcn_raw_buffer_store_b128(v0, dstR[q], voff, doff, 17);
      __builtin_amdgcn_raw_buffer_store_b128(v1, dstR[q], voff, doff1, 17);
    }
    if (n == 0 && wid == 0) STAMP(1);
    if (n >= 1) {
      WAIT_V(16);
      signal(n - 1); if (wid == 0) STAMP(1 + n);
    }
    fetch(n + 3);
  }
  WAIT_V(0); signal(31);
  if (wid == 0) STAMP(33);
}

__device__ __forceinline__ i32x8 ld_frag(LAS unsigned char* a) {
  const i32x4 lo = *(const LAS i32x4*)a, hi = *(const LAS i32x4*)(a + 1024);
  return __builtin_shufflevector(lo, hi, 0, 1, 2, 3, 4, 5, 6, 7);
}

__device__ __forceinline__ i32x8 ld_frag_tr(LAS unsigned char* a) {
  const i32x2 q0 = __builtin_amdgcn_ds_read_tr8_b64_v2i32((LAS i32x2*)a), q1 = __builtin_amdgcn_ds_read_tr8_b64_v2i32((LAS i32x2*)(a + 1024));
  const i32x2 q2 = __builtin_amdgcn_ds_read_tr8_b64_v2i32((LAS i32x2*)(a + 8192)), q3 = __builtin_amdgcn_ds_read_tr8_b64_v2i32((LAS i32x2*)(a + 9216));
  return (i32x8){q0.x, q0.y, q1.x, q1.y, q2.x, q2.y, q3.x, q3.y};
}

__device__ __forceinline__ void gemm_role(const Params& p, LAS unsigned char* lds, unsigned call, int wid, int lane) {
  const int g = blockIdx.x - NCOMM, L = 30 * (g & 7) + (g >> 3);
  const int tj = (L < 192) ? ((L & 63) >> 2) : (L - 192) / 3, tr = (L < 192) ? ((L >> 6) * 4 + (L & 3)) : 12 + (L - 192) % 3;
  const int wr = wid >> 2, wc = wid & 3;
  const int rank = p.rank & 7;
  char* myws = p.ws[rank];
  const int U = 17 * tr, UX = (tr == 14) ? 255 : U + 8;
  const __amdgpu_buffer_rsrc_t opR = mk_rsrc(myws + (size_t)(call & 1u) * BUF_BYTES, BUF_BYTES);
  const __amdgpu_buffer_rsrc_t xR = mk_rsrc(p.x, SHARD_BYTES), wR = mk_rsrc(p.w, SHARD_BYTES);
  const unsigned voff = (unsigned)lane * 16u;
  volatile LAS unsigned* ctrl = (volatile LAS unsigned*)(lds + L_CTRL);
  unsigned* tmo = (unsigned*)(myws + TMO_OFF);

  const unsigned sA0a = (unsigned)(U * 2 + wid) * 1024u;
  const unsigned sA0x = (wid < 6) ? (unsigned)(U * 2 + 12 + wid) * 1024u : (unsigned)(UX * 2 + wid - 6) * 1024u;
  const unsigned sA1 = (unsigned)((U + 9) * 2 + wid) * 1024u;
  const unsigned sB = OPER_BYTES + (unsigned)(tj * 32 + wid) * 1024u;
  const unsigned vx = (unsigned)((lane & 15) * 512 + (lane >> 4) * 16);
  const unsigned vw = (unsigned)((lane >> 3) * 4096 + (((lane & 7) ^ (((lane >> 4) & 3) | ((wid & 2) << 1))) * 16));
  const unsigned lA0a = (unsigned)((U + (wid >> 1)) * 8192 + (wid & 1) * 64);
  const unsigned lA0x = (unsigned)(((wid < 6) ? U + 8 : UX) * 8192 + (wid & 1) * 64);
  const unsigned lA1 = lA0a + 9u * 8192u;
  const unsigned lB = (unsigned)(wid * 8 * 4096 + tj * 256);
  const unsigned lw = (unsigned)wid * 1024u;
#define GLDS(ldsoff, soff) __builtin_amdgcn_raw_ptr_buffer_load_lds(opR, (LAS void*)(lds + (ldsoff)), 16, voff, (soff), 0, 17)
#define GLDX(ldsoff, soff) __builtin_amdgcn_raw_ptr_buffer_load_lds(xR, (LAS void*)(lds + (ldsoff)), 16, vx, (soff), 0, 0)
#define GLDW(ldsoff, soff) __builtin_amdgcn_raw_ptr_buffer_load_lds(wR, (LAS void*)(lds + (ldsoff)), 16, vw, (soff), 0, 0)
#define STAGE_A0(LOC, b, kb) do { if (LOC) { GLDX(L_A0 + (b) * L_A0B + lw, lA0a + (kb) * 128u); GLDX(L_A0 + (b) * L_A0B + lw + 8192, lA0a + 32768u + (kb) * 128u); \
                                             if (wr) GLDX(L_A0 + (b) * L_A0B + lw + 12288, lA0x + (kb) * 128u); } \
                                  else { GLDS(L_A0 + (b) * L_A0B + lw, (kb) + sA0a); GLDS(L_A0 + (b) * L_A0B + lw + 8192, (kb) + sA0a + 8192); \
                                         if (wr) GLDS(L_A0 + (b) * L_A0B + lw + 12288, (kb) + sA0x); } } while (0)
#define STAGE_A1(LOC, b, kb) do { if (LOC) { GLDX(L_A1 + (b) * L_A1B + lw, lA1 + (kb) * 128u); GLDX(L_A1 + (b) * L_A1B + lw + 8192, lA1 + 32768u + (kb) * 128u); } \
                                  else { GLDS(L_A1 + (b) * L_A1B + lw, (kb) + sA1); GLDS(L_A1 + (b) * L_A1B + lw + 8192, (kb) + sA1 + 8192); } } while (0)
#define STAGE_B(LOC, b, bj, kb) do { if (LOC) { GLDW(L_B + (b) * L_BB + (bj) * 16384 + lw, lB + (kb) * 524288u + (bj) * 128); \
                                                GLDW(L_B + (b) * L_BB + (bj) * 16384 + lw + 8192, lB + (kb) * 524288u + (bj) * 128 + 262144u); } \
                                     else { GLDS(L_B + (b) * L_BB + (bj) * 16384 + lw, (kb) + sB + (bj) * 16384); \
                                            GLDS(L_B + (b) * L_BB + (bj) * 16384 + lw + 8192, (kb) + sB + (bj) * 16384 + 8192); } } while (0)
#define WAIT_TILE do { if (wr) WAIT_V(7); else WAIT_V(6); } while (0)

  LAS unsigned char* rA0 = lds + L_A0 + wr * 10240 + lane * 16;
  LAS unsigned char* rA1 = lds + L_A1 + wr * 8192 + lane * 16;
  LAS unsigned char* rB = lds + L_B + wc * 4096 + lane * 16;
  const int fsw = (((lane & 15) >> 2) & 3) | (((lane >> 4) & 1) << 2);
  LAS unsigned char* rBl0 = lds + L_B + ((lane >> 4) * 16 + ((lane & 15) >> 1)) * 128 + (((wc * 2) ^ fsw) * 16) + (lane & 1) * 8;
  LAS unsigned char* rBl1 = lds + L_B + ((lane >> 4) * 16 + ((lane & 15) >> 1)) * 128 + (((wc * 2 + 1) ^ fsw) * 16) + (lane & 1) * 8;
#define LDA0(b) do { _Pragma("unroll") for (int m = 0; m < 5; ++m) At[m] = ld_frag(rA0 + (b) * L_A0B + m * 2048); } while (0)
#define LDA1(b) do { _Pragma("unroll") for (int m = 0; m < 4; ++m) At[m] = ld_frag(rA1 + (b) * L_A1B + m * 2048); } while (0)
#define LDB(LOC, dst, b, bj) do { if (LOC) { dst[0] = ld_frag_tr(rBl0 + (b) * L_BB + (bj) * 16384); dst[1] = ld_frag_tr(rBl1 + (b) * L_BB + (bj) * 16384); } \
    else { _Pragma("unroll") for (int n = 0; n < 2; ++n) dst[n] = ld_frag(rB + (b) * L_BB + (bj) * 16384 + n * 2048); } } while (0)
#define MMA0(bj, Bt) do { __builtin_amdgcn_s_setprio(1); _Pragma("unroll") for (int m = 0; m < 5; ++m) _Pragma("unroll") for (int n = 0; n < 2; ++n) \
    acc0[bj][m][n] = __builtin_amdgcn_mfma_scale_f32_16x16x128_f8f6f4(Bt[n], At[m], acc0[bj][m][n], 0, 0, 0, 0, 0, 0); __builtin_amdgcn_s_setprio(0); } while (0)
#define MMA1(bj, Bt) do { __builtin_amdgcn_s_setprio(1); _Pragma("unroll") for (int m = 0; m < 4; ++m) _Pragma("unroll") for (int n = 0; n < 2; ++n) \
    acc1[bj][m][n] = __builtin_amdgcn_mfma_scale_f32_16x16x128_f8f6f4(Bt[n], At[m], acc1[bj][m][n], 0, 0, 0, 0, 0, 0); __builtin_amdgcn_s_setprio(0); } while (0)

  f32x4 acc0[2][5][2], acc1[2][4][2];
#pragma unroll
  for (int b = 0; b < 2; ++b)
#pragma unroll
    for (int n = 0; n < 2; ++n) {
#pragma unroll
      for (int m = 0; m < 5; ++m) acc0[b][m][n] = (f32x4){0.f, 0.f, 0.f, 0.f};
#pragma unroll
      for (int m = 0; m < 4; ++m) acc1[b][m][n] = (f32x4){0.f, 0.f, 0.f, 0.f};
    }
  i32x8 At[5], B0[2], B1[2];

  const char* fpa = myws + MBOX_OFF + (size_t)((tr * 16 + tj) * 128 + (lane < 3 ? lane : 0) * 8);
  u32x2 pa = {0u, 0u};
  unsigned done = 0xFu << (4 * rank), bad = 0;
  auto issue_poll = [&]() {
    asm volatile("global_load_dwordx2 %0, %1, off sc0 sc1" : "=&v"(pa) : "v"(fpa) : "memory");
  };
  auto ready_mask = [&]() -> unsigned {
    const unsigned mk = pa[0], tg = pa[1];
    const unsigned mv = (tg == call) ? mk : 0u;
    const unsigned m = (unsigned)__builtin_amdgcn_readlane(mv, 0) & (unsigned)__builtin_amdgcn_readlane(mv, 1) & (unsigned)__builtin_amdgcn_readlane(mv, 2);
    return m & ~done;
  };
  auto decide = [&](int slot) {
    asm volatile("" : "+v"(pa));
    unsigned m = ready_mask();
    if (m == 0u) {
      const unsigned long long t0 = __builtin_amdgcn_s_memrealtime();
      while (m == 0u) {
        if (bad == 0u) {
          issue_poll();
          asm volatile("s_waitcnt vmcnt(0)" : "+v"(pa) :: "memory");
          m = ready_mask();
          if (m == 0u) {
            const unsigned long long dt = __builtin_amdgcn_s_memrealtime() - t0;
            if (dt > 100000ull) {
              const unsigned tv = (unsigned)__builtin_amdgcn_readfirstlane(__hip_atomic_load(tmo, __ATOMIC_RELAXED, __HIP_MEMORY_SCOPE_AGENT));
              if (tv == call || dt > DEADLINE_TICKS) {
                if (lane == 0) { __hip_atomic_store(tmo, call, __ATOMIC_RELAXED, __HIP_MEMORY_SCOPE_AGENT); ctrl[2] = 1u; }
                bad = 1u;
              }
              __builtin_amdgcn_s_sleep(64);
            }
            __builtin_amdgcn_s_sleep(POLL_SLEEP);
          }
        } else {
          m = ~done;
        }
        m = (unsigned)__builtin_amdgcn_readfirstlane(m);
        bad = (unsigned)__builtin_amdgcn_readfirstlane(bad);
      }
    }
    const unsigned ks = (unsigned)__builtin_ctz(m);
    done = (unsigned)__builtin_amdgcn_readfirstlane(done | (1u << ks));
    if (lane == 0) ctrl[slot] = ks;
    issue_poll();
  };
#define READ_D(slot) ((unsigned)__builtin_amdgcn_readfirstlane(ctrl[slot]) * (512u * 1024u))

  if (wid == 0) { if (lane == 0) ctrl[2] = 0u; issue_poll(); }
  STAGE_B(1, 0, 0, 0u); STAGE_A0(1, 0, 0u); STAGE_B(1, 0, 1, 0u); STAGE_A1(1, 0, 0u);
  if (wr == 1) BAR;
  WAIT_V(4); BAR;
  STAGE_B(1, 1, 0, 1u); STAGE_A0(1, 1, 1u); STAGE_B(1, 1, 1, 1u);
  WAIT_TILE; BAR;

#define TRIP(RL, S1L, SNL, more) do { \
    unsigned kb2 = 2u, kb3 = 3u; \
      \
    if (!(SNL) && wid == 0 && (more)) decide(0); \
    LDB(RL, B0, 0, 0); SCHED; LDA0(0); STAGE_A1(S1L, 1, kb1); \
    WAIT_L(10); BAR; WAIT_L(0); MMA0(0, B0); BAR; SCHED; \
      \
    if (!(SNL) && (more)) kb2 = READ_D(0); \
    LDB(RL, B1, 0, 1); if (more) STAGE_B(SNL, 0, 0, kb2); \
    BAR; WAIT_L(0); MMA0(1, B1); BAR; \
      \
    LDA1(0); if (more) STAGE_A0(SNL, 0, kb2); \
    BAR; WAIT_L(0); MMA1(0, B0); BAR; SCHED; \
      \
    if (more) { STAGE_B(SNL, 0, 1, kb2); WAIT_TILE; } else { WAIT_V(0); } \
    BAR; MMA1(1, B1); BAR; \
      \
    if (!(SNL) && wid == 0 && (more)) decide(1); \
    LDB(RL, B0, 1, 0); SCHED; LDA0(1); if (more) STAGE_A1(SNL, 0, kb2); \
    WAIT_L(10); BAR; WAIT_L(0); MMA0(0, B0); BAR; SCHED; \
      \
    if (!(SNL) && (more)) kb3 = READ_D(1); \
    LDB(RL, B1, 1, 1); if (more) STAGE_B(SNL, 1, 0, kb3); \
    BAR; WAIT_L(0); MMA0(1, B1); BAR; \
      \
    LDA1(1); if (more) STAGE_A0(SNL, 1, kb3); \
    BAR; WAIT_L(0); MMA1(0, B0); BAR; SCHED; \
      \
    if (more) { STAGE_B(SNL, 1, 1, kb3); WAIT_TILE; } \
    BAR; MMA1(1, B1); BAR; \
    kb1 = kb3; } while (0)
  unsigned kb1 = 1u;
#pragma unroll 1
  for (int it = 0; it < 16; ++it) { const bool more = it < 15, rl = it < 2, snl = it == 0; TRIP(rl, rl, snl, more); }
  if (wr == 0) BAR;
  if (wid == 0) STAMP(40);

  const float scale = p.sx[0] * p.sw[0];
  const bool poison = (ctrl[2] != 0u) || (__hip_atomic_load(tmo, __ATOMIC_RELAXED, __HIP_MEMORY_SCOPE_AGENT) == call);
  const float qn = __builtin_nanf("");
  const int fr = lane & 15, fq = lane >> 4;
  float* obase = p.out + (size_t)fr * 4096 + tj * 256 + wc * 32 + fq * 4;
  auto put = [&](const f32x4& a, int unit, int bj, int n) {
    f32x4 y = a * scale;
#pragma unroll
    for (int e = 0; e < 4; ++e) y[e] = (y[e] < 0.f) ? 0.f : y[e];
    if (poison) y = (f32x4){qn, qn, qn, qn};
    *(f32x4*)(obase + (size_t)unit * 16 * 4096 + bj * 128 + n * 16) = y;
  };
#pragma unroll
  for (int m = 0; m < 5; ++m) {
    const int slot = wr * 5 + m;
    if (slot < 9 || tr == 14) {
      const int unit = (slot < 9) ? U + slot : 255;
#pragma unroll
      for (int bj = 0; bj < 2; ++bj)
#pragma unroll
        for (int n = 0; n < 2; ++n) put(acc0[bj][m][n], unit, bj, n);
    }
  }
#pragma unroll
  for (int m = 0; m < 4; ++m) {
    const int unit = U + 9 + wr * 4 + m;
#pragma unroll
    for (int bj = 0; bj < 2; ++bj)
#pragma unroll
      for (int n = 0; n < 2; ++n) put(acc1[bj][m][n], unit, bj, n);
  }
}
}

__global__ void __launch_bounds__(512, 2) fused_gemm_ag_kernel(Params p) {
  extern __shared__ __attribute__((aligned(16))) unsigned char smem_dyn[];
  LAS unsigned char* lds = (LAS unsigned char*)smem_dyn;
  const int wid = __builtin_amdgcn_readfirstlane(threadIdx.x >> 6), lane = threadIdx.x & 63;
  unsigned* ctr = (unsigned*)(p.ws[p.rank & 7] + CTR_OFF) + blockIdx.x * 16;
  if (wid == 0) STAMP(0);
  unsigned call;
  if (blockIdx.x < NCOMM) comm_role(p, lds, ctr, call, wid, lane);
  else { call = (unsigned)__builtin_amdgcn_readfirstlane(ld_sys(ctr)) + 1u;
    gemm_role(p, lds, call, wid, lane);
  }
  if (threadIdx.x == 0) __hip_atomic_store(ctr, call, __ATOMIC_RELAXED, __HIP_MEMORY_SCOPE_SYSTEM);
}

extern "C" void kernel_launch(int rank, int nranks, ncclComm_t comm,
                              hipStream_t stream,
                              void* const* d_in, const int* in_sizes, int n_in,
                              void* d_out, int out_size,
                              const int* in_dtypes, int out_dtype,
                              void* const* d_out_all,
                              void* const* d_ws_all, size_t ws_bytes) {
  (void)comm; (void)in_sizes; (void)n_in; (void)out_size; (void)in_dtypes; (void)out_dtype; (void)d_out_all;
  if (nranks != NRANK || (unsigned)rank >= (unsigned)NRANK || ws_bytes < (size_t)WS_USED) {
    fprintf(stderr, "unsupported configuration\n"); return;
  }
  HIP_CHECK(hipFuncSetAttribute((const void*)fused_gemm_ag_kernel, hipFuncAttributeMaxDynamicSharedMemorySize, (int)LDS_BYTES));
  Params p;
  p.x = (const unsigned char*)d_in[0]; p.w = (const unsigned char*)d_in[1];
  p.sx = (const float*)d_in[2]; p.sw = (const float*)d_in[3];
  p.out = (float*)d_out; p.rank = rank;
  for (int i = 0; i < NRANK; ++i) p.ws[i] = (char*)d_ws_all[i];
  hipLaunchKernelGGL(fused_gemm_ag_kernel, dim3(NWG), dim3(512), LDS_BYTES, stream, p);
  HIP_CHECK(hipGetLastError());
}
```

```cpp
#include <hip/hip_runtime.h>
#include <rccl/rccl.h>
#include <cstdio>
#include <cstdint>

#define HIP_CHECK(expr) do { hipError_t _e = (expr); if (_e != hipSuccess) { \
    fprintf(stderr, "HIP error %s at %s:%d\n", hipGetErrorString(_e), __FILE__, __LINE__); \
    return; } } while (0)

#define LAS __attribute__((address_space(3)))
typedef int i32x8 __attribute__((ext_vector_type(8)));
typedef int i32x4 __attribute__((ext_vector_type(4)));
typedef int i32x2 __attribute__((ext_vector_type(2)));
typedef unsigned u32x4 __attribute__((ext_vector_type(4)));
typedef unsigned u32x2 __attribute__((ext_vector_type(2)));
typedef float f32x4 __attribute__((ext_vector_type(4)));
typedef unsigned short u16x2 __attribute__((ext_vector_type(2)));

namespace {
constexpr int NRANK = 8;
constexpr int NCOMM = 16, NGEMM = 240, NWG = NCOMM + NGEMM;
constexpr unsigned OPER_BYTES = 16u << 20;
constexpr unsigned BUF_BYTES = 2u * OPER_BYTES;
constexpr unsigned FLAG_OFF = 2u * BUF_BYTES;
constexpr unsigned FLAG_BYTES = 32u * 32u * 4u * 2u;
constexpr unsigned CTR_OFF = FLAG_OFF + (64u << 10);
constexpr unsigned TMO_OFF = FLAG_OFF + (128u << 10);
constexpr unsigned TICK_OFF = FLAG_OFF + (160u << 10);
constexpr unsigned MBOX_OFF = FLAG_OFF + (192u << 10);
constexpr unsigned WS_USED = FLAG_OFF + (256u << 10);
constexpr unsigned SHARD_BYTES = 2u << 20;
#define W_LEAD 0
#define TICK_Q 4
#define RELAY_SLEEP 1
#define POLL_SLEEP 2
constexpr unsigned long long DEADLINE_TICKS = 50000000ull;

constexpr unsigned L_A0 = 0, L_A0B = 20480;
constexpr unsigned L_A1 = 40960, L_A1B = 16384;
constexpr unsigned L_B = 73728, L_BB = 32768;
constexpr unsigned L_CTRL = 139264;
constexpr unsigned C_RING = 1024;
constexpr unsigned LDS_BYTES = L_CTRL + 64;

struct Params {
  const unsigned char* x; const unsigned char* w; const float* sx; const float* sw; float* out;
  char* ws[NRANK]; int rank;
};
#define STAMP(i) do { } while (0)

__device__ __forceinline__ __amdgpu_buffer_rsrc_t mk_rsrc(const void* p, unsigned bytes) {
  return __builtin_amdgcn_make_buffer_rsrc((void*)p, 0, bytes, 0x00020000);
}
__device__ __forceinline__ unsigned ld_sys(const unsigned* p) {
  return __hip_atomic_load(p, __ATOMIC_RELAXED, __HIP_MEMORY_SCOPE_SYSTEM);
}
#define CFENCE asm volatile("" ::: "memory")
#define WAIT_V(n) asm volatile("s_waitcnt vmcnt(" #n ")" ::: "memory")
#define WAIT_L(n) asm volatile("s_waitcnt lgkmcnt(" #n ")" ::: "memory")
#define BAR do { CFENCE; __builtin_amdgcn_s_barrier(); CFENCE; } while (0)
#define SCHED __builtin_amdgcn_sched_barrier(0)

__device__ __forceinline__ void comm_role(const Params& p, LAS unsigned char* lds, const unsigned* ctr, unsigned& call_out, int wid, int lane) {
  const int c = blockIdx.x;
  const int rank = p.rank & 7;
  LAS unsigned* cnt = (LAS unsigned*)lds;
  volatile LAS unsigned* ctl = (volatile LAS unsigned*)lds;
  if (threadIdx.x < 18) ctl[threadIdx.x] = (threadIdx.x >= 8 && threadIdx.x < 8 + TICK_Q) ? (unsigned)(c + 16 * (threadIdx.x - 8)) : (threadIdx.x == 16 ? (unsigned)TICK_Q : 0u);
  if (wid >= 4) {
    const unsigned call = (unsigned)__builtin_amdgcn_readfirstlane(ld_sys(ctr)) + 1u;
    call_out = call;
    WAIT_L(0); BAR;
    if (wid == 7) return;
    if (wid == 6) {
      unsigned* tick = (unsigned*)(p.ws[rank] + TICK_OFF) + (call & 1u) * 64;
      if (c == 0 && lane == 0) __hip_atomic_store((unsigned*)(p.ws[rank] + TICK_OFF) + ((call & 1u) ^ 1u) * 64, 0u, __ATOMIC_RELAXED, __HIP_MEMORY_SCOPE_AGENT);
      unsigned posted = TICK_Q;
      for (;;) {
        while (posted - (unsigned)__builtin_amdgcn_readfirstlane(ctl[17]) >= (unsigned)TICK_Q) __builtin_amdgcn_s_sleep(1);
        unsigned t = 0;
        if (lane == 0) t = __hip_atomic_fetch_add(tick, 1u, __ATOMIC_RELAXED, __HIP_MEMORY_SCOPE_AGENT);
        t = (unsigned)__builtin_amdgcn_readfirstlane(t) + 16u * TICK_Q;
        if (lane == 0) { ctl[8 + (posted & 7u)] = t; ctl[16] = posted + 1u; }
        ++posted;
        if (t >= 512u) break;
      }
      return;
    }
    const int e = 2 * c + (wid - 4);
    char* myws = p.ws[rank];
    const char* fp = myws + FLAG_OFF + (size_t)((e * 32 + (lane & 31)) * 8);
    int ctr_r, ctj, slot; bool tgt;
    if (e < 16) { ctr_r = (lane < 16) ? e : e - 1; ctj = lane & 15; slot = (lane < 16) ? 0 : 1; tgt = lane < 32 && (unsigned)ctr_r < 15u; }
    else { ctr_r = lane; ctj = e - 16; slot = 2; tgt = lane < 15; }
    unsigned long long* mb = (unsigned long long*)(myws + MBOX_OFF + (size_t)(((tgt ? ctr_r : 0) * 16 + ctj) * 128 + slot * 8));
    unsigned* tmo = (unsigned*)(myws + TMO_OFF);
    const unsigned short c16 = (unsigned short)call; const u16x2 cc = {c16, c16};
    unsigned last = 0;
    const unsigned long long t0 = __builtin_amdgcn_s_memrealtime();
    for (;;) {
      const unsigned long long fv = __hip_atomic_load((const unsigned long long*)fp, __ATOMIC_RELAXED, __HIP_MEMORY_SCOPE_SYSTEM);
      u16x2 a = __builtin_bit_cast(u16x2, (unsigned)fv), b = __builtin_bit_cast(u16x2, (unsigned)(fv >> 32));
      a -= cc; b -= cc;
      const unsigned dd = __builtin_bit_cast(unsigned, a) | __builtin_bit_cast(unsigned, b);
      const unsigned m = (unsigned)__builtin_amdgcn_ballot_w64((dd & 0xFFFEFFFEu) == 0u || ((lane & 31) >> 2) == rank);
      if (m != last) {
        if (tgt) __hip_atomic_store(mb, ((unsigned long long)call << 32) | m, __ATOMIC_RELAXED, __HIP_MEMORY_SCOPE_SYSTEM);
        last = m;
        if (m == 0xFFFFFFFFu) break;
      }
      if (__builtin_amdgcn_s_memrealtime() - t0 > DEADLINE_TICKS) {
        if (lane == 0) __hip_atomic_store(tmo, call, __ATOMIC_RELAXED, __HIP_MEMORY_SCOPE_AGENT);
        break;
      }
      __builtin_amdgcn_s_sleep(RELAY_SLEEP);
    }
    return;
  }
  LAS unsigned char* ring = lds + C_RING + wid * 8192;
  const __amdgpu_buffer_rsrc_t xR = mk_rsrc(p.x, SHARD_BYTES), wR = mk_rsrc(p.w, SHARD_BYTES);
  const unsigned lp_w = (unsigned)((lane >> 2) * 4096 + (lane & 3) * 16), lp_x = (unsigned)((lane & 15) * 512 + (lane >> 4) * 16);
  auto blk = [&](unsigned t, int& o, int& eb, int& s, int& q4) {
    const int idx = (int)((t < 512u ? t : 511u) >> 2);
    const int j = idx - W_LEAD; int k;
    q4 = (int)(t & 3u);
    if (j < 0) { o = 1; k = idx; }
    else if (j < 2 * (64 - W_LEAD)) { o = j & 1; k = (j >> 1) + (o ? W_LEAD : 0); }
    else { o = 0; k = j - (64 - W_LEAD); }
    s = k >> 4; eb = ((k & 15) + 2 * rank) & 15;
  };
  auto fetch = [&](unsigned t, unsigned slot) {
    int o, eb, s, q4; blk(t, o, eb, s, q4);
    const int u0 = eb * 16 + 4 * q4;
    LAS unsigned char* dst = ring + (slot & 3u) * 2048;
    if (o) {
      const unsigned so = (unsigned)((s * 128 + 32 * wid) * 4096 + u0 * 16);
      __builtin_amdgcn_raw_ptr_buffer_load_lds(wR, (LAS void*)dst, 16, lp_w, so, 0, 0);
      __builtin_amdgcn_raw_ptr_buffer_load_lds(wR, (LAS void*)(dst + 1024), 16, lp_w, so + 16u * 4096u, 0, 0);
    } else {
      const unsigned so = (unsigned)((u0 + wid) * 8192 + s * 128);
      __builtin_amdgcn_raw_ptr_buffer_load_lds(xR, (LAS void*)dst, 16, lp_x, so, 0, 0);
      __builtin_amdgcn_raw_ptr_buffer_load_lds(xR, (LAS void*)(dst + 1024), 16, lp_x, so + 64u, 0, 0);
    }
  };
  const unsigned craw = ld_sys(ctr);
  unsigned t0 = (unsigned)c, t1 = (unsigned)c + 16u, tprev = 0;
  fetch(t0, 0); fetch(t1, 1);
  const unsigned call = (unsigned)__builtin_amdgcn_readfirstlane(craw) + 1u;
  asm volatile("" :: "s"(call));
  call_out = call;
  __amdgpu_buffer_rsrc_t dstR[7];
  {
    const int rot = (c * 4 + wid) % 7;
#pragma unroll
    for (int q = 0; q < 7; ++q) { const int pr = (rank + 1 + (q + rot) % 7) & 7; dstR[q] = mk_rsrc(p.ws[pr], FLAG_OFF); }
  }
  char* flp = p.ws[0];
#pragma unroll
  for (int q = 1; q < 8; ++q) flp = ((lane & 7) == q) ? p.ws[q] : flp;
  flp += FLAG_OFF;
  asm volatile("" :: "v"(flp));
  const unsigned vo_w = (unsigned)((lane >> 5) * 2048 + ((lane >> 4) & 1) * 256 + (lane & 15) * 16), vo_x = (unsigned)lane * 16u;
  const unsigned dcommon = (call & 1u) * BUF_BYTES;
  const unsigned db_w = dcommon + OPER_BYTES + (unsigned)((wid >> 1) * 1024 + (wid & 1) * 512), db_x = dcommon + (unsigned)wid * 2048u;
  const unsigned rd_w = (unsigned)(size_t)ring + (unsigned)((((lane >> 4) & 1) * 16 + ((lane & 15) >> 1)) * 64 + (lane >> 5) * 16 + (lane & 1) * 8);
  const unsigned rd_x = (unsigned)(size_t)ring + (unsigned)lane * 16u;
  WAIT_L(0); BAR;

  auto signal = [&](unsigned i, unsigned t) {
    unsigned old = 0;
    if (lane == 0) old = __hip_atomic_fetch_add(cnt + (i & 7u), 1u, __ATOMIC_RELAXED, __HIP_MEMORY_SCOPE_WORKGROUP);
    old = __builtin_amdgcn_readfirstlane(old);
    if (old == 3u) {
      int o, eb, s, q4; blk(t, o, eb, s, q4);
      const unsigned idx = (unsigned)((((o * 16 + eb) * 32) + rank * 4 + s) * 4 + q4);
      if (lane < 8 && lane != rank) __hip_atomic_store((unsigned short*)flp + idx, (unsigned short)call, __ATOMIC_RELAXED, __HIP_MEMORY_SCOPE_SYSTEM);
      if (lane == 0) { ctl[i & 7u] = 0u; __hip_atomic_fetch_add(cnt + 17, 1u, __ATOMIC_RELAXED, __HIP_MEMORY_SCOPE_WORKGROUP); }
    }
  };
  unsigned i = 0;
#pragma unroll 1
  for (; t0 < 512u; ++i) {
    if (i == 0) WAIT_V(2); else WAIT_V(16);
    int o, eb, s, q4; blk(t0, o, eb, s, q4);
    u32x4 v0, v1;
    if (o) {
      const unsigned a = rd_w + (i & 3u) * 2048u;
      u32x2 a0, a1, b0, b1;
      asm volatile("ds_read_b64_tr_b8 %0, %4\n\tds_read_b64_tr_b8 %1, %4 offset:512\n\t"
                   "ds_read_b64_tr_b8 %2, %4 offset:32\n\tds_read_b64_tr_b8 %3, %4 offset:544\n\ts_waitcnt lgkmcnt(0)"
                   : "=&v"(a0), "=&v"(a1), "=&v"(b0), "=&v"(b1) : "v"(a) : "memory");
      v0 = (u32x4){a0.x, a0.y, a1.x, a1.y}; v1 = (u32x4){b0.x, b0.y, b1.x, b1.y};
    } else {
      const unsigned a = rd_x + (i & 3u) * 2048u;
      asm volatile("ds_read_b128 %0, %2\n\tds_read_b128 %1, %2 offset:1024\n\ts_waitcnt lgkmcnt(0)"
                   : "=&v"(v0), "=&v"(v1) : "v"(a) : "memory");
    }
    const unsigned voff = o ? vo_w : vo_x;
    const unsigned doff = (o ? db_w : db_x) + (unsigned)(((rank * 4 + s) * 256 + eb * 16 + 4 * q4) * 2048);
    const unsigned doff1 = doff + (o ? 4096u : 1024u);
#pragma unroll
    for (int q = 0; q < 7; ++q) {
      __builtin_amdgcn_raw_buffer_store_b128(v0, dstR[q], voff, doff, 17);
      __builtin_amdgcn_raw_buffer_store_b128(v1, dstR[q], voff, doff1, 17);
    }
    if (i == 0 && wid == 0) STAMP(1);
    if (i >= 1) {
      WAIT_V(16);
      signal(i - 1, tprev);
    }
    unsigned t2 = 512u;
    if (t1 < 512u) {
      while ((unsigned)__builtin_amdgcn_readfirstlane(ctl[16]) <= i + 2u) __builtin_amdgcn_s_sleep(1);
      t2 = (unsigned)__builtin_amdgcn_readfirstlane(ctl[8 + ((i + 2u) & 7u)]);
    }
    fetch(t2, i + 2u);
    tprev = t0; t0 = t1; t1 = t2;
  }
  WAIT_V(0); signal(i - 1, tprev);
  if (wid == 0) STAMP(33);
}

__device__ __forceinline__ i32x8 ld_frag(LAS unsigned char* a) {
  const i32x4 lo = *(const LAS i32x4*)a, hi = *(const LAS i32x4*)(a + 1024);
  return __builtin_shufflevector(lo, hi, 0, 1, 2, 3, 4, 5, 6, 7);
}

__device__ __forceinline__ i32x8 ld_frag_tr(LAS unsigned char* a) {
  const i32x2 q0 = __builtin_amdgcn_ds_read_tr8_b64_v2i32((LAS i32x2*)a), q1 = __builtin_amdgcn_ds_read_tr8_b64_v2i32((LAS i32x2*)(a + 1024));
  const i32x2 q2 = __builtin_amdgcn_ds_read_tr8_b64_v2i32((LAS i32x2*)(a + 8192)), q3 = __builtin_amdgcn_ds_read_tr8_b64_v2i32((LAS i32x2*)(a + 9216));
  return (i32x8){q0.x, q0.y, q1.x, q1.y, q2.x, q2.y, q3.x, q3.y};
}

__device__ __forceinline__ void gemm_role(const Params& p, LAS unsigned char* lds, unsigned call, int wid, int lane) {
  const int g = blockIdx.x - NCOMM, L = 30 * (g & 7) + (g >> 3);
  const int tj = (L < 192) ? ((L & 63) >> 2) : (L - 192) / 3, tr = (L < 192) ? ((L >> 6) * 4 + (L & 3)) : 12 + (L - 192) % 3;
  const int wr = wid >> 2, wc = wid & 3;
  const int rank = p.rank & 7;
  char* myws = p.ws[rank];
  const int U = 17 * tr, UX = (tr == 14) ? 255 : U + 8;
  const __amdgpu_buffer_rsrc_t opR = mk_rsrc(myws + (size_t)(call & 1u) * BUF_BYTES, BUF_BYTES);
  const __amdgpu_buffer_rsrc_t xR = mk_rsrc(p.x, SHARD_BYTES), wR = mk_rsrc(p.w, SHARD_BYTES);
  const unsigned voff = (unsigned)lane * 16u;
  volatile LAS unsigned* ctrl = (volatile LAS unsigned*)(lds + L_CTRL);
  unsigned* tmo = (unsigned*)(myws + TMO_OFF);

  const unsigned sA0a = (unsigned)(U * 2 + wid) * 1024u;
  const unsigned sA0x = (wid < 6) ? (unsigned)(U * 2 + 12 + wid) * 1024u : (unsigned)(UX * 2 + wid - 6) * 1024u;
  const unsigned sA1 = (unsigned)((U + 9) * 2 + wid) * 1024u;
  const unsigned sB = OPER_BYTES + (unsigned)(tj * 32 + wid) * 1024u;
  const unsigned vx = (unsigned)((lane & 15) * 512 + (lane >> 4) * 16);
  const unsigned vw = (unsigned)((lane >> 3) * 4096 + (((lane & 7) ^ (((lane >> 4) & 3) | ((wid & 2) << 1))) * 16));
  const unsigned lA0a = (unsigned)((U + (wid >> 1)) * 8192 + (wid & 1) * 64);
  const unsigned lA0x = (unsigned)(((wid < 6) ? U + 8 : UX) * 8192 + (wid & 1) * 64);
  const unsigned lA1 = lA0a + 9u * 8192u;
  const unsigned lB = (unsigned)(wid * 8 * 4096 + tj * 256);
  const unsigned lw = (unsigned)wid * 1024u;
#define GLDS(ldsoff, soff) __builtin_amdgcn_raw_ptr_buffer_load_lds(opR, (LAS void*)(lds + (ldsoff)), 16, voff, (soff), 0, 17)
#define GLDX(ldsoff, soff) __builtin_amdgcn_raw_ptr_buffer_load_lds(xR, (LAS void*)(lds + (ldsoff)), 16, vx, (soff), 0, 0)
#define GLDW(ldsoff, soff) __builtin_amdgcn_raw_ptr_buffer_load_lds(wR, (LAS void*)(lds + (ldsoff)), 16, vw, (soff), 0, 0)
#define STAGE_A0(LOC, b, kb) do { if (LOC) { GLDX(L_A0 + (b) * L_A0B + lw, lA0a + (kb) * 128u); GLDX(L_A0 + (b) * L_A0B + lw + 8192, lA0a + 32768u + (kb) * 128u); \
                                             if (wr) GLDX(L_A0 + (b) * L_A0B + lw + 12288, lA0x + (kb) * 128u); } \
                                  else { GLDS(L_A0 + (b) * L_A0B + lw, (kb) + sA0a); GLDS(L_A0 + (b) * L_A0B + lw + 8192, (kb) + sA0a + 8192); \
                                         if (wr) GLDS(L_A0 + (b) * L_A0B + lw + 12288, (kb) + sA0x); } } while (0)
#define STAGE_A1(LOC, b, kb) do { if (LOC) { GLDX(L_A1 + (b) * L_A1B + lw, lA1 + (kb) * 128u); GLDX(L_A1 + (b) * L_A1B + lw + 8192, lA1 + 32768u + (kb) * 128u); } \
                                  else { GLDS(L_A1 + (b) * L_A1B + lw, (kb) + sA1); GLDS(L_A1 + (b) * L_A1B + lw + 8192, (kb) + sA1 + 8192); } } while (0)
#define STAGE_B(LOC, b, bj, kb) do { if (LOC) { GLDW(L_B + (b) * L_BB + (bj) * 16384 + lw, lB + (kb) * 524288u + (bj) * 128); \
                                                GLDW(L_B + (b) * L_BB + (bj) * 16384 + lw + 8192, lB + (kb) * 524288u + (bj) * 128 + 262144u); } \
                                     else { GLDS(L_B + (b) * L_BB + (bj) * 16384 + lw, (kb) + sB + (bj) * 16384); \
                                            GLDS(L_B + (b) * L_BB + (bj) * 16384 + lw + 8192, (kb) + sB + (bj) * 16384 + 8192); } } while (0)
#define WAIT_TILE do { if (wr) WAIT_V(7); else WAIT_V(6); } while (0)

  const bool use5 = (wr == 0) || (tr == 14);
  LAS unsigned char* rA0 = lds + L_A0 + wr * 10240 + lane * 16;
  LAS unsigned char* rA1 = lds + L_A1 + wr * 8192 + lane * 16;
  LAS unsigned char* rB = lds + L_B + wc * 4096 + lane * 16;
  const int fsw = (((lane & 15) >> 2) & 3) | (((lane >> 4) & 1) << 2);
  LAS unsigned char* rBl0 = lds + L_B + ((lane >> 4) * 16 + ((lane & 15) >> 1)) * 128 + (((wc * 2) ^ fsw) * 16) + (lane & 1) * 8;
  LAS unsigned char* rBl1 = lds + L_B + ((lane >> 4) * 16 + ((lane & 15) >> 1)) * 128 + (((wc * 2 + 1) ^ fsw) * 16) + (lane & 1) * 8;
#define LDA0(b) do { _Pragma("unroll") for (int m = 0; m < 4; ++m) At[m] = ld_frag(rA0 + (b) * L_A0B + m * 2048); } while (0)
#define LDA04(b) do { At[4] = ld_frag(rA0 + (b) * L_A0B + 4 * 2048); } while (0)
#define WAIT_HALF(more) do { if (more) { if (wr) WAIT_V(9); else WAIT_V(8); } else WAIT_V(4); } while (0)
#define LDA1(b) do { _Pragma("unroll") for (int m = 0; m < 4; ++m) At[m] = ld_frag(rA1 + (b) * L_A1B + m * 2048); } while (0)
#define LDB(LOC, dst, b, bj) do { if (LOC) { dst[0] = ld_frag_tr(rBl0 + (b) * L_BB + (bj) * 16384); dst[1] = ld_frag_tr(rBl1 + (b) * L_BB + (bj) * 16384); } \
    else { _Pragma("unroll") for (int n = 0; n < 2; ++n) dst[n] = ld_frag(rB + (b) * L_BB + (bj) * 16384 + n * 2048); } } while (0)
#define MMA0(bj, Bt) do { __builtin_amdgcn_s_setprio(1); _Pragma("unroll") for (int m = 0; m < 4; ++m) _Pragma("unroll") for (int n = 0; n < 2; ++n) \
    acc0[bj][m][n] = __builtin_amdgcn_mfma_scale_f32_16x16x128_f8f6f4(Bt[n], At[m], acc0[bj][m][n], 0, 0, 0, 0, 0, 0); \
    if (use5) { _Pragma("unroll") for (int n = 0; n < 2; ++n) \
    acc0[bj][4][n] = __builtin_amdgcn_mfma_scale_f32_16x16x128_f8f6f4(Bt[n], At[4], acc0[bj][4][n], 0, 0, 0, 0, 0, 0); } __builtin_amdgcn_s_setprio(0); } while (0)
#define MMA1(bj, Bt) do { __builtin_amdgcn_s_setprio(1); _Pragma("unroll") for (int m = 0; m < 4; ++m) _Pragma("unroll") for (int n = 0; n < 2; ++n) \
    acc1[bj][m][n] = __builtin_amdgcn_mfma_scale_f32_16x16x128_f8f6f4(Bt[n], At[m], acc1[bj][m][n], 0, 0, 0, 0, 0, 0); __builtin_amdgcn_s_setprio(0); } while (0)

  f32x4 acc0[2][5][2], acc1[2][4][2];
#pragma unroll
  for (int b = 0; b < 2; ++b)
#pragma unroll
    for (int n = 0; n < 2; ++n) {
#pragma unroll
      for (int m = 0; m < 5; ++m) acc0[b][m][n] = (f32x4){0.f, 0.f, 0.f, 0.f};
#pragma unroll
      for (int m = 0; m < 4; ++m) acc1[b][m][n] = (f32x4){0.f, 0.f, 0.f, 0.f};
    }
  i32x8 At[5], B0[2], B1[2];

  const char* fpa = myws + MBOX_OFF + (size_t)((tr * 16 + tj) * 128 + (lane < 3 ? lane : 0) * 8);
  u32x2 pa = {0u, 0u};
  unsigned done = 0xFu << (4 * rank), bad = 0;
  auto issue_poll = [&]() {
    asm volatile("global_load_dwordx2 %0, %1, off sc0 sc1" : "=&v"(pa) : "v"(fpa) : "memory");
  };
  auto ready_mask = [&]() -> unsigned {
    const unsigned mk = pa[0], tg = pa[1];
    const unsigned mv = (tg == call) ? mk : 0u;
    const unsigned m = (unsigned)__builtin_amdgcn_readlane(mv, 0) & (unsigned)__builtin_amdgcn_readlane(mv, 1) & (unsigned)__builtin_amdgcn_readlane(mv, 2);
    return m & ~done;
  };
  auto decide = [&](int slot) {
    asm volatile("" : "+v"(pa));
    unsigned m = ready_mask();
    if (m == 0u) {
      const unsigned long long t0 = __builtin_amdgcn_s_memrealtime();
      while (m == 0u) {
        if (bad == 0u) {
          issue_poll();
          asm volatile("s_waitcnt vmcnt(0)" : "+v"(pa) :: "memory");
          m = ready_mask();
          if (m == 0u) {
            const unsigned long long dt = __builtin_amdgcn_s_memrealtime() - t0;
            if (dt > 100000ull) {
              const unsigned tv = (unsigned)__builtin_amdgcn_readfirstlane(__hip_atomic_load(tmo, __ATOMIC_RELAXED, __HIP_MEMORY_SCOPE_AGENT));
              if (tv == call || dt > DEADLINE_TICKS) {
                if (lane == 0) { __hip_atomic_store(tmo, call, __ATOMIC_RELAXED, __HIP_MEMORY_SCOPE_AGENT); ctrl[2] = 1u; }
                bad = 1u;
              }
              __builtin_amdgcn_s_sleep(64);
            }
            __builtin_amdgcn_s_sleep(POLL_SLEEP);
          }
        } else {
          m = ~done;
        }
        m = (unsigned)__builtin_amdgcn_readfirstlane(m);
        bad = (unsigned)__builtin_amdgcn_readfirstlane(bad);
      }
    }
    const unsigned ks = (unsigned)__builtin_ctz(m);
    done = (unsigned)__builtin_amdgcn_readfirstlane(done | (1u << ks));
    if (lane == 0) ctrl[slot] = ks;
    issue_poll();
  };
#define READ_D(slot) ((unsigned)__builtin_amdgcn_readfirstlane(ctrl[slot]) * (512u * 1024u))

  const float scale = __builtin_bit_cast(float, __builtin_amdgcn_readfirstlane(__builtin_bit_cast(int, p.sx[0] * p.sw[0])));
  if (wid == 0) { if (lane == 0) ctrl[2] = 0u; issue_poll(); }
  STAGE_B(1, 0, 0, 0u); STAGE_A0(1, 0, 0u); STAGE_B(1, 0, 1, 0u); STAGE_A1(1, 0, 0u);
  if (wr == 1) BAR;
  WAIT_V(4); BAR;
  STAGE_B(1, 1, 0, 1u); STAGE_A0(1, 1, 1u); STAGE_B(1, 1, 1, 1u);
  WAIT_TILE; BAR;

  LDB(1, B0, 0, 0); LDA04(0);
#define TRIP(RL, S1L, SNL, more) do { \
    unsigned kb2 = 2u, kb3 = 3u; \
      \
    if (!(SNL) && wid == 0 && (more)) decide(0); \
    if (wid == 0 && !(more)) asm volatile("global_load_dword %0, %1, off sc1" : "=&v"(pa[0]) : "v"(tmo) : "memory");     \
    LDA0(0); STAGE_A1(S1L, 1, kb1); \
    BAR; WAIT_L(0); MMA0(0, B0); BAR; SCHED; \
      \
    if (!(SNL) && (more)) kb2 = READ_D(0); \
    LDB(RL, B1, 0, 1); if (more) STAGE_B(SNL, 0, 0, kb2); \
    BAR; WAIT_L(0); MMA0(1, B1); BAR; \
      \
    LDA1(0); if (more) STAGE_A0(SNL, 0, kb2); \
    WAIT_HALF(more); BAR; WAIT_L(0); MMA1(0, B0); BAR; SCHED; \
      \
    LDB(RL, B0, 1, 0); LDA04(1); \
    if (more) { STAGE_B(SNL, 0, 1, kb2); WAIT_TILE; } else { WAIT_V(0); } \
    BAR; MMA1(1, B1); BAR; \
      \
    if (!(SNL) && wid == 0 && (more)) decide(1); \
    if (wid == 0 && !(more)) { asm volatile("" : "+v"(pa)); if (lane == 0 && pa[0] == call) ctrl[2] = 1u; } \
    LDA0(1); if (more) STAGE_A1(SNL, 0, kb2); \
    BAR; WAIT_L(0); MMA0(0, B0); BAR; SCHED; \
      \
    if (!(SNL) && (more)) kb3 = READ_D(1); \
    LDB(RL, B1, 1, 1); if (more) STAGE_B(SNL, 1, 0, kb3); \
    BAR; WAIT_L(0); MMA0(1, B1); BAR; \
      \
    LDA1(1); if (more) { STAGE_A0(SNL, 1, kb3); WAIT_HALF(true); } \
    BAR; WAIT_L(0); MMA1(0, B0); BAR; SCHED; \
      \
    if (more) { LDB(SNL, B0, 0, 0); LDA04(0); STAGE_B(SNL, 1, 1, kb3); WAIT_TILE; } \
    BAR; MMA1(1, B1); BAR; \
    kb1 = kb3; } while (0)
  unsigned kb1 = 1u;
#pragma unroll 1
  for (int gi = 0; gi < 2; ++gi) { const bool snl = gi == 0; TRIP(1, 1, snl, true); }
#pragma unroll 1
  for (int it = 2; it < 15; ++it) TRIP(0, 0, 0, true);
  TRIP(0, 0, 0, false);
  if (wr == 0) BAR;
  if (wid == 0) STAMP(40);

  const float escale = (ctrl[2] != 0u) ? __builtin_nanf("") : scale;
  const int fr = lane & 15, fq = lane >> 4;
  LAS unsigned char* ewr = lds + (wr * 16 + fr) * 1040 + (wc * 32 + fq * 4) * 4;
  LAS unsigned char* erd = lds + (wid * 4) * 1040 + lane * 16;
  float* obase = p.out + (size_t)((wid * 4) & 15) * 4096 + tj * 256 + lane * 4;
  auto pass = [&](int j, const f32x4& a00, const f32x4& a01, const f32x4& a10, const f32x4& a11, int unit, bool live) {
    const f32x4* src[4] = {&a00, &a01, &a10, &a11};
#pragma unroll
    for (int q = 0; q < 4; ++q) {
      f32x4 y = *src[q] * escale;
#pragma unroll
      for (int e = 0; e < 4; ++e) y[e] = (y[e] < 0.f) ? 0.f : y[e];
      *(LAS f32x4*)(ewr + (j & 3) * 33280 + (q >> 1) * 512 + (q & 1) * 64) = y;
    }
    WAIT_L(0); BAR;
    if (live) {
      f32x4 r[4];
#pragma unroll
      for (int i = 0; i < 4; ++i) r[i] = *(const LAS f32x4*)(erd + (j & 3) * 33280 + i * 1040);
#pragma unroll
      for (int i = 0; i < 4; ++i) *(f32x4*)(obase + (size_t)(unit * 16 + i) * 4096) = r[i];
    }
  };
#pragma unroll
  for (int m = 0; m < 5; ++m) {
    const int slot = wr * 5 + m;
    pass(m, acc0[0][m][0], acc0[0][m][1], acc0[1][m][0], acc0[1][m][1], (slot < 9) ? U + slot : 255, slot < 9 || tr == 14);
  }
#pragma unroll
  for (int m = 0; m < 4; ++m) pass(5 + m, acc1[0][m][0], acc1[0][m][1], acc1[1][m][0], acc1[1][m][1], U + 9 + wr * 4 + m, true);
}
}

__global__ void __launch_bounds__(512, 2) fused_gemm_ag_kernel(Params p) {
  extern __shared__ __attribute__((aligned(16))) unsigned char smem_dyn[];
  LAS unsigned char* lds = (LAS unsigned char*)smem_dyn;
  const int wid = __builtin_amdgcn_readfirstlane(threadIdx.x >> 6), lane = threadIdx.x & 63;
  unsigned* ctr = (unsigned*)(p.ws[p.rank & 7] + CTR_OFF) + blockIdx.x * 16;
  if (wid == 0) STAMP(0);
  unsigned call;
  if (blockIdx.x < NCOMM) comm_role(p, lds, ctr, call, wid, lane);
  else { call = (unsigned)__builtin_amdgcn_readfirstlane(ld_sys(ctr)) + 1u;
    gemm_role(p, lds, call, wid, lane);
  }
  if (threadIdx.x == 0) __hip_atomic_store(ctr, call, __ATOMIC_RELAXED, __HIP_MEMORY_SCOPE_SYSTEM);
}

extern "C" void kernel_launch(int rank, int nranks, ncclComm_t comm,
                              hipStream_t stream,
                              void* const* d_in, const int* in_sizes, int n_in,
                              void* d_out, int out_size,
                              const int* in_dtypes, int out_dtype,
                              void* const* d_out_all,
                              void* const* d_ws_all, size_t ws_bytes) {
  (void)comm; (void)in_sizes; (void)n_in; (void)out_size; (void)in_dtypes; (void)out_dtype; (void)d_out_all;
  if (nranks != NRANK || (unsigned)rank >= (unsigned)NRANK || ws_bytes < (size_t)WS_USED) {
    fprintf(stderr, "unsupported configuration\n"); return;
  }
  HIP_CHECK(hipFuncSetAttribute((const void*)fused_gemm_ag_kernel, hipFuncAttributeMaxDynamicSharedMemorySize, (int)LDS_BYTES));
  Params p;
  p.x = (const unsigned char*)d_in[0]; p.w = (const unsigned char*)d_in[1];
  p.sx = (const float*)d_in[2]; p.sw = (const float*)d_in[3];
  p.out = (float*)d_out; p.rank = rank;
  for (int i = 0; i < NRANK; ++i) p.ws[i] = (char*)d_ws_all[i];
  hipLaunchKernelGGL(fused_gemm_ag_kernel, dim3(NWG), dim3(512), LDS_BYTES, stream, p);
  HIP_CHECK(hipGetLastError());
}
```

```cpp
#include <hip/hip_runtime.h>
#include <rccl/rccl.h>
#include <cstdio>
#include <cstdint>

#define HIP_CHECK(expr) do { hipError_t _e = (expr); if (_e != hipSuccess) { \
    fprintf(stderr, "HIP error %s at %s:%d\n", hipGetErrorString(_e), __FILE__, __LINE__); \
    return; } } while (0)

#define LAS __attribute__((address_space(3)))
typedef int i32x8 __attribute__((ext_vector_type(8)));
typedef int i32x4 __attribute__((ext_vector_type(4)));
typedef int i32x2 __attribute__((ext_vector_type(2)));
typedef unsigned u32x4 __attribute__((ext_vector_type(4)));
typedef unsigned u32x2 __attribute__((ext_vector_type(2)));
typedef float f32x4 __attribute__((ext_vector_type(4)));
typedef unsigned short u16x2 __attribute__((ext_vector_type(2)));

namespace {
constexpr int NRANK = 8;
constexpr int NCOMM = 16, NGEMM = 240, NWG = NCOMM + NGEMM;
constexpr unsigned OPER_BYTES = 16u << 20;
constexpr unsigned BUF_BYTES = 2u * OPER_BYTES;
constexpr unsigned FLAG_OFF = 2u * BUF_BYTES;
constexpr unsigned FLAG_BYTES = 32u * 32u * 4u * 2u;
constexpr unsigned CTR_OFF = FLAG_OFF + (64u << 10);
constexpr unsigned TMO_OFF = FLAG_OFF + (128u << 10);
constexpr unsigned TICK_OFF = FLAG_OFF + (160u << 10);
constexpr unsigned MBOX_OFF = FLAG_OFF + (192u << 10);
constexpr unsigned WS_USED = FLAG_OFF + (256u << 10);
constexpr unsigned SHARD_BYTES = 2u << 20;
constexpr int TICK_Q = 4;
constexpr unsigned long long DEADLINE_TICKS = 50000000ull;

constexpr unsigned L_A0 = 0, L_A0B = 20480;
constexpr unsigned L_A1 = 40960, L_A1B = 16384;
constexpr unsigned L_B = 73728, L_BB = 32768;
constexpr unsigned L_CTRL = 139264;
constexpr unsigned C_RING = 1024;
constexpr unsigned LDS_BYTES = L_CTRL + 64;

struct Params {
  const unsigned char* x; const unsigned char* w; const float* sx; const float* sw; float* out;
  char* ws[NRANK]; int rank;
};

__device__ __forceinline__ __amdgpu_buffer_rsrc_t mk_rsrc(const void* p, unsigned bytes) {
  return __builtin_amdgcn_make_buffer_rsrc((void*)p, 0, bytes, 0x00020000);
}
__device__ __forceinline__ unsigned ld_sys(const unsigned* p) {
  return __hip_atomic_load(p, __ATOMIC_RELAXED, __HIP_MEMORY_SCOPE_SYSTEM);
}
#define CFENCE asm volatile("" ::: "memory")
#define WAIT_V(n) asm volatile("s_waitcnt vmcnt(" #n ")" ::: "memory")
#define WAIT_L(n) asm volatile("s_waitcnt lgkmcnt(" #n ")" ::: "memory")
#define BAR do { CFENCE; __builtin_amdgcn_s_barrier(); CFENCE; } while (0)
#define SCHED __builtin_amdgcn_sched_barrier(0)

__device__ __forceinline__ void comm_role(const Params& p, LAS unsigned char* lds, const unsigned* ctr, unsigned& call_out, int wid, int lane) {
  const int c = blockIdx.x;
  const int rank = p.rank & 7;
  LAS unsigned* cnt = (LAS unsigned*)lds;
  volatile LAS unsigned* ctl = (volatile LAS unsigned*)lds;
  if (threadIdx.x < 18) ctl[threadIdx.x] = (threadIdx.x >= 8 && threadIdx.x < 8 + TICK_Q) ? (unsigned)(c + 16 * (threadIdx.x - 8)) : (threadIdx.x == 16 ? (unsigned)TICK_Q : 0u);
  if (wid >= 4) {
    const unsigned call = (unsigned)__builtin_amdgcn_readfirstlane(ld_sys(ctr)) + 1u;
    call_out = call;
    WAIT_L(0); BAR;
    if (wid == 7) return;
    if (wid == 6) {
      unsigned* tick = (unsigned*)(p.ws[rank] + TICK_OFF) + (call & 1u) * 64;
      if (c == 0 && lane == 0) __hip_atomic_store((unsigned*)(p.ws[rank] + TICK_OFF) + ((call & 1u) ^ 1u) * 64, 0u, __ATOMIC_RELAXED, __HIP_MEMORY_SCOPE_AGENT);
      unsigned posted = TICK_Q;
      for (;;) {
        while (posted - (unsigned)__builtin_amdgcn_readfirstlane(ctl[17]) >= (unsigned)TICK_Q) __builtin_amdgcn_s_sleep(1);
        unsigned t = 0;
        if (lane == 0) t = __hip_atomic_fetch_add(tick, 1u, __ATOMIC_RELAXED, __HIP_MEMORY_SCOPE_AGENT);
        t = (unsigned)__builtin_amdgcn_readfirstlane(t) + 16u * TICK_Q;
        if (lane == 0) { ctl[8 + (posted & 7u)] = t; ctl[16] = posted + 1u; }
        ++posted;
        if (t >= 512u) break;
      }
      return;
    }
    const int e = 2 * c + (wid - 4);
    char* myws = p.ws[rank];
    const char* fp = myws + FLAG_OFF + (size_t)((e * 32 + (lane & 31)) * 8);
    int ctr_r, ctj, slot; bool tgt;
    if (e < 16) { ctr_r = (lane < 16) ? e : e - 1; ctj = lane & 15; slot = (lane < 16) ? 0 : 1; tgt = lane < 32 && (unsigned)ctr_r < 15u; }
    else { ctr_r = lane; ctj = e - 16; slot = 2; tgt = lane < 15; }
    unsigned long long* mb = (unsigned long long*)(myws + MBOX_OFF + (size_t)(((tgt ? ctr_r : 0) * 16 + ctj) * 128 + slot * 8));
    unsigned* tmo = (unsigned*)(myws + TMO_OFF);
    const unsigned short c16 = (unsigned short)call; const u16x2 cc = {c16, c16};
    unsigned last = 0;
    const unsigned long long t0 = __builtin_amdgcn_s_memrealtime();
    for (;;) {
      const unsigned long long fv = __hip_atomic_load((const unsigned long long*)fp, __ATOMIC_RELAXED, __HIP_MEMORY_SCOPE_SYSTEM);
      u16x2 a = __builtin_bit_cast(u16x2, (unsigned)fv), b = __builtin_bit_cast(u16x2, (unsigned)(fv >> 32));
      a -= cc; b -= cc;
      const unsigned dd = __builtin_bit_cast(unsigned, a) | __builtin_bit_cast(unsigned, b);
      const unsigned m = (unsigned)__builtin_amdgcn_ballot_w64((dd & 0xFFFEFFFEu) == 0u || ((lane & 31) >> 2) == rank);
      if (m != last) {
        if (tgt) __hip_atomic_store(mb, ((unsigned long long)call << 32) | m, __ATOMIC_RELAXED, __HIP_MEMORY_SCOPE_SYSTEM);
        last = m;
        if (m == 0xFFFFFFFFu) break;
      }
      if (__builtin_amdgcn_s_memrealtime() - t0 > DEADLINE_TICKS) {
        if (lane == 0) __hip_atomic_store(tmo, call, __ATOMIC_RELAXED, __HIP_MEMORY_SCOPE_AGENT);
        break;
      }
      __builtin_amdgcn_s_sleep(1);
    }
    return;
  }
  LAS unsigned char* ring = lds + C_RING + wid * 8192;
  const __amdgpu_buffer_rsrc_t xR = mk_rsrc(p.x, SHARD_BYTES), wR = mk_rsrc(p.w, SHARD_BYTES);
  const unsigned lp_w = (unsigned)((lane >> 2) * 4096 + (lane & 3) * 16), lp_x = (unsigned)((lane & 15) * 512 + (lane >> 4) * 16);
  auto blk = [&](unsigned t, int& o, int& eb, int& s, int& q4) {
    const int idx = (int)((t < 512u ? t : 511u) >> 2);
    q4 = (int)(t & 3u); o = idx & 1; s = idx >> 5;
    eb = (((idx >> 1) & 15) + 2 * rank) & 15;
  };
  auto fetch = [&](unsigned t, unsigned slot) {
    int o, eb, s, q4; blk(t, o, eb, s, q4);
    const int u0 = eb * 16 + 4 * q4;
    LAS unsigned char* dst = ring + (slot & 3u) * 2048;
    if (o) {
      const unsigned so = (unsigned)((s * 128 + 32 * wid) * 4096 + u0 * 16);
      __builtin_amdgcn_raw_ptr_buffer_load_lds(wR, (LAS void*)dst, 16, lp_w, so, 0, 0);
      __builtin_amdgcn_raw_ptr_buffer_load_lds(wR, (LAS void*)(dst + 1024), 16, lp_w, so + 16u * 4096u, 0, 0);
    } else {
      const unsigned so = (unsigned)((u0 + wid) * 8192 + s * 128);
      __builtin_amdgcn_raw_ptr_buffer_load_lds(xR, (LAS void*)dst, 16, lp_x, so, 0, 0);
      __builtin_amdgcn_raw_ptr_buffer_load_lds(xR, (LAS void*)(dst + 1024), 16, lp_x, so + 64u, 0, 0);
    }
  };
  const unsigned craw = ld_sys(ctr);
  unsigned t0 = (unsigned)c, t1 = (unsigned)c + 16u, tprev = 0;
  fetch(t0, 0); fetch(t1, 1);
  const unsigned call = (unsigned)__builtin_amdgcn_readfirstlane(craw) + 1u;
  asm volatile("" :: "s"(call));
  call_out = call;
  __amdgpu_buffer_rsrc_t dstR[7];
  {
    const int rot = (c * 4 + wid) % 7;
#pragma unroll
    for (int q = 0; q < 7; ++q) { const int pr = (rank + 1 + (q + rot) % 7) & 7; dstR[q] = mk_rsrc(p.ws[pr], FLAG_OFF); }
  }
  unsigned flo = 0, fhi = 0;
#pragma unroll
  for (int q = 0; q < 8; ++q) {
    const unsigned long long a = (unsigned long long)(p.ws[q] + FLAG_OFF);
    unsigned lo = (unsigned)a, hi = (unsigned)(a >> 32);
    asm volatile("" : "+s"(lo), "+s"(hi));
    flo = ((lane & 7) == q) ? lo : flo; fhi = ((lane & 7) == q) ? hi : fhi;
  }
  typedef __attribute__((address_space(1))) unsigned short gu16;
  gu16* flp = (gu16*)(((unsigned long long)fhi << 32) | flo);
  const unsigned vo_w = (unsigned)((lane >> 5) * 2048 + ((lane >> 4) & 1) * 256 + (lane & 15) * 16), vo_x = (unsigned)lane * 16u;
  const unsigned dcommon = (call & 1u) * BUF_BYTES;
  const unsigned db_w = dcommon + OPER_BYTES + (unsigned)((wid >> 1) * 1024 + (wid & 1) * 512), db_x = dcommon + (unsigned)wid * 2048u;
  const unsigned rd_w = (unsigned)(size_t)ring + (unsigned)((((lane >> 4) & 1) * 16 + ((lane & 15) >> 1)) * 64 + (lane >> 5) * 16 + (lane & 1) * 8);
  const unsigned rd_x = (unsigned)(size_t)ring + (unsigned)lane * 16u;
  WAIT_L(0); BAR;

  auto signal = [&](unsigned i, unsigned t) {
    unsigned old = 0;
    if (lane == 0) old = __hip_atomic_fetch_add(cnt + (i & 7u), 1u, __ATOMIC_RELAXED, __HIP_MEMORY_SCOPE_WORKGROUP);
    old = __builtin_amdgcn_readfirstlane(old);
    if (old == 3u) {
      int o, eb, s, q4; blk(t, o, eb, s, q4);
      const unsigned idx = (unsigned)((((o * 16 + eb) * 32) + rank * 4 + s) * 4 + q4);
      if (lane < 8 && lane != rank) __hip_atomic_store(flp + idx, (unsigned short)call, __ATOMIC_RELAXED, __HIP_MEMORY_SCOPE_SYSTEM);
      if (lane == 0) { ctl[i & 7u] = 0u; __hip_atomic_fetch_add(cnt + 17, 1u, __ATOMIC_RELAXED, __HIP_MEMORY_SCOPE_WORKGROUP); }
    }
  };
  unsigned i = 0;
#pragma unroll 1
  for (; t0 < 512u; ++i) {
    if (i == 0) WAIT_V(2); else WAIT_V(16);
    int o, eb, s, q4; blk(t0, o, eb, s, q4);
    u32x4 v0, v1;
    if (o) {
      const unsigned a = rd_w + (i & 3u) * 2048u;
      u32x2 a0, a1, b0, b1;
      asm volatile("ds_read_b64_tr_b8 %0, %4\n\tds_read_b64_tr_b8 %1, %4 offset:512\n\t"
                   "ds_read_b64_tr_b8 %2, %4 offset:32\n\tds_read_b64_tr_b8 %3, %4 offset:544\n\ts_waitcnt lgkmcnt(0)"
                   : "=&v"(a0), "=&v"(a1), "=&v"(b0), "=&v"(b1) : "v"(a) : "memory");
      v0 = (u32x4){a0.x, a0.y, a1.x, a1.y}; v1 = (u32x4){b0.x, b0.y, b1.x, b1.y};
    } else {
      const unsigned a = rd_x + (i & 3u) * 2048u;
      asm volatile("ds_read_b128 %0, %2\n\tds_read_b128 %1, %2 offset:1024\n\ts_waitcnt lgkmcnt(0)"
                   : "=&v"(v0), "=&v"(v1) : "v"(a) : "memory");
    }
    const unsigned voff = o ? vo_w : vo_x;
    const unsigned doff = (o ? db_w : db_x) + (unsigned)(((rank * 4 + s) * 256 + eb * 16 + 4 * q4) * 2048);
    const unsigned doff1 = doff + (o ? 4096u : 1024u);
#pragma unroll
    for (int q = 0; q < 7; ++q) {
      __builtin_amdgcn_raw_buffer_store_b128(v0, dstR[q], voff, doff, 17);
      __builtin_amdgcn_raw_buffer_store_b128(v1, dstR[q], voff, doff1, 17);
    }
    if (i >= 1) {
      WAIT_V(16);
      signal(i - 1, tprev);
    }
    unsigned t2 = 512u;
    if (t1 < 512u) {
      while ((unsigned)__builtin_amdgcn_readfirstlane(ctl[16]) <= i + 2u) __builtin_amdgcn_s_sleep(1);
      t2 = (unsigned)__builtin_amdgcn_readfirstlane(ctl[8 + ((i + 2u) & 7u)]);
    }
    fetch(t2, i + 2u);
    tprev = t0; t0 = t1; t1 = t2;
  }
  WAIT_V(0); signal(i - 1, tprev);
}

__device__ __forceinline__ i32x8 ld_frag(LAS unsigned char* a) {
  const i32x4 lo = *(const LAS i32x4*)a, hi = *(const LAS i32x4*)(a + 1024);
  return __builtin_shufflevector(lo, hi, 0, 1, 2, 3, 4, 5, 6, 7);
}

__device__ __forceinline__ i32x8 ld_frag_tr(LAS unsigned char* a) {
  const i32x2 q0 = __builtin_amdgcn_ds_read_tr8_b64_v2i32((LAS i32x2*)a), q1 = __builtin_amdgcn_ds_read_tr8_b64_v2i32((LAS i32x2*)(a + 1024));
  const i32x2 q2 = __builtin_amdgcn_ds_read_tr8_b64_v2i32((LAS i32x2*)(a + 8192)), q3 = __builtin_amdgcn_ds_read_tr8_b64_v2i32((LAS i32x2*)(a + 9216));
  return (i32x8){q0.x, q0.y, q1.x, q1.y, q2.x, q2.y, q3.x, q3.y};
}

__device__ __forceinline__ void gemm_role(const Params& p, LAS unsigned char* lds, unsigned call, int wid, int lane) {
  const int g = blockIdx.x - NCOMM;
  const int L = 30 * (g & 7) + (g >> 3);
  const int tj = (L < 192) ? ((L & 63) >> 2) : (L - 192) / 3, tr = (L < 192) ? ((L >> 6) * 4 + (L & 3)) : 12 + (L - 192) % 3;
  const int wr = wid >> 2, wc = wid & 3;
  const int rank = p.rank & 7;
  char* myws = p.ws[rank];
  const int U = 17 * tr, UX = (tr == 14) ? 255 : U + 8;
  const __amdgpu_buffer_rsrc_t opR = mk_rsrc(myws + (size_t)(call & 1u) * BUF_BYTES, BUF_BYTES);
  const __amdgpu_buffer_rsrc_t xR = mk_rsrc(p.x, SHARD_BYTES), wR = mk_rsrc(p.w, SHARD_BYTES);
  const unsigned voff = (unsigned)lane * 16u;
  volatile LAS unsigned* ctrl = (volatile LAS unsigned*)(lds + L_CTRL);
  unsigned* tmo = (unsigned*)(myws + TMO_OFF);
  const __amdgpu_buffer_rsrc_t tmoR = mk_rsrc(tmo, 64);

  const unsigned sA0a = (unsigned)(U * 2 + wid) * 1024u;
  const unsigned sA0x = (wid < 6) ? (unsigned)(U * 2 + 12 + wid) * 1024u : (unsigned)(UX * 2 + wid - 6) * 1024u;
  const unsigned sA1 = (unsigned)((U + 9) * 2 + wid) * 1024u;
  const unsigned sB = OPER_BYTES + (unsigned)((tj * 16 + 4 * (wid >> 2) + ((wid >> 1) & 1)) * 2048 + (wid & 1) * 1024);
  const unsigned vx = (unsigned)((lane & 15) * 512 + (lane >> 4) * 16);
  const int vch = (lane & 7) ^ (((lane >> 4) & 3) | ((wid & 2) << 1));
  const unsigned vw = (unsigned)((lane >> 3) * 4096 + (vch >> 1) * 64 + (vch & 1) * 16);
  const unsigned lA0a = (unsigned)((U + (wid >> 1)) * 8192 + (wid & 1) * 64);
  const unsigned lA0x = (unsigned)(((wid < 6) ? U + 8 : UX) * 8192 + (wid & 1) * 64);
  const unsigned lA1 = lA0a + 9u * 8192u;
  const unsigned lB = (unsigned)(wid * 8 * 4096 + tj * 256);
  const unsigned lw = (unsigned)wid * 1024u;
#define GLDS(ldsoff, soff) __builtin_amdgcn_raw_ptr_buffer_load_lds(opR, (LAS void*)(lds + (ldsoff)), 16, voff, (soff), 0, 17)
#define GLDX(ldsoff, soff) __builtin_amdgcn_raw_ptr_buffer_load_lds(xR, (LAS void*)(lds + (ldsoff)), 16, vx, (soff), 0, 0)
#define GLDW(ldsoff, soff) __builtin_amdgcn_raw_ptr_buffer_load_lds(wR, (LAS void*)(lds + (ldsoff)), 16, vw, (soff), 0, 0)
#define STAGE_A0(LOC, b, kb) do { if (LOC) { GLDX(L_A0 + (b) * L_A0B + lw, lA0a + (kb) * 128u); GLDX(L_A0 + (b) * L_A0B + lw + 8192, lA0a + 32768u + (kb) * 128u); \
                                             if (wr) GLDX(L_A0 + (b) * L_A0B + lw + 12288, lA0x + (kb) * 128u); } \
                                  else { GLDS(L_A0 + (b) * L_A0B + lw, (kb) + sA0a); GLDS(L_A0 + (b) * L_A0B + lw + 8192, (kb) + sA0a + 8192); \
                                         if (wr) GLDS(L_A0 + (b) * L_A0B + lw + 12288, (kb) + sA0x); } } while (0)
#define STAGE_A1(LOC, b, kb) do { if (LOC) { GLDX(L_A1 + (b) * L_A1B + lw, lA1 + (kb) * 128u); GLDX(L_A1 + (b) * L_A1B + lw + 8192, lA1 + 32768u + (kb) * 128u); } \
                                  else { GLDS(L_A1 + (b) * L_A1B + lw, (kb) + sA1); GLDS(L_A1 + (b) * L_A1B + lw + 8192, (kb) + sA1 + 8192); } } while (0)
#define STAGE_B(LOC, b, bj, kb) do { if (LOC) { GLDW(L_B + (b) * L_BB + (bj) * 16384 + lw, lB + (kb) * 524288u + (bj) * 32); \
                                                GLDW(L_B + (b) * L_BB + (bj) * 16384 + lw + 8192, lB + (kb) * 524288u + (bj) * 32 + 262144u); } \
                                     else { GLDS(L_B + (b) * L_BB + (bj) * 16384 + lw, (kb) + sB + (bj) * 4096); \
                                            GLDS(L_B + (b) * L_BB + (bj) * 16384 + lw + 8192, (kb) + sB + (bj) * 4096 + 16384); } } while (0)
#define WAIT_TILE do { if (wr) WAIT_V(7); else WAIT_V(6); } while (0)

  const bool use5 = (wr == 0) || (tr == 14);
  LAS unsigned char* rA0 = lds + L_A0 + wr * 10240 + lane * 16;
  LAS unsigned char* rA1 = lds + L_A1 + wr * 8192 + lane * 16;
  LAS unsigned char* rB = lds + L_B + wc * 4096 + lane * 16;
  const int fsw = (((lane & 15) >> 2) & 3) | (((lane >> 4) & 1) << 2);
  LAS unsigned char* rBl0 = lds + L_B + ((lane >> 4) * 16 + ((lane & 15) >> 1)) * 128 + (((wc * 2) ^ fsw) * 16) + (lane & 1) * 8;
  LAS unsigned char* rBl1 = lds + L_B + ((lane >> 4) * 16 + ((lane & 15) >> 1)) * 128 + (((wc * 2 + 1) ^ fsw) * 16) + (lane & 1) * 8;
#define LDA0(b) do { _Pragma("unroll") for (int m = 0; m < 4; ++m) At[m] = ld_frag(rA0 + (b) * L_A0B + m * 2048); } while (0)
#define LDA04(b) do { At[4] = ld_frag(rA0 + (b) * L_A0B + 4 * 2048); } while (0)
#define WAIT_HALF(more) do { if (more) { if (wr) WAIT_V(9); else WAIT_V(8); } else WAIT_V(4); } while (0)
#define LDA1(b) do { _Pragma("unroll") for (int m = 0; m < 4; ++m) At[m] = ld_frag(rA1 + (b) * L_A1B + m * 2048); } while (0)
#define LDB(LOC, dst, b, bj) do { if (LOC) { dst[0] = ld_frag_tr(rBl0 + (b) * L_BB + (bj) * 16384); dst[1] = ld_frag_tr(rBl1 + (b) * L_BB + (bj) * 16384); } \
    else { _Pragma("unroll") for (int n = 0; n < 2; ++n) dst[n] = ld_frag(rB + (b) * L_BB + (bj) * 16384 + n * 2048); } } while (0)
#define MMA0(bj, Bt) do { __builtin_amdgcn_s_setprio(1); _Pragma("unroll") for (int m = 0; m < 4; ++m) _Pragma("unroll") for (int n = 0; n < 2; ++n) \
    acc0[bj][m][n] = __builtin_amdgcn_mfma_scale_f32_16x16x128_f8f6f4(Bt[n], At[m], acc0[bj][m][n], 0, 0, 0, 0, 0, 0); \
    if (use5) { _Pragma("unroll") for (int n = 0; n < 2; ++n) \
    acc0[bj][4][n] = __builtin_amdgcn_mfma_scale_f32_16x16x128_f8f6f4(Bt[n], At[4], acc0[bj][4][n], 0, 0, 0, 0, 0, 0); } __builtin_amdgcn_s_setprio(0); } while (0)
#define MMA1(bj, Bt) do { __builtin_amdgcn_s_setprio(1); _Pragma("unroll") for (int m = 0; m < 4; ++m) _Pragma("unroll") for (int n = 0; n < 2; ++n) \
    acc1[bj][m][n] = __builtin_amdgcn_mfma_scale_f32_16x16x128_f8f6f4(Bt[n], At[m], acc1[bj][m][n], 0, 0, 0, 0, 0, 0); __builtin_amdgcn_s_setprio(0); } while (0)

  f32x4 acc0[2][5][2], acc1[2][4][2];
#pragma unroll
  for (int b = 0; b < 2; ++b)
#pragma unroll
    for (int n = 0; n < 2; ++n) {
#pragma unroll
      for (int m = 0; m < 5; ++m) acc0[b][m][n] = (f32x4){0.f, 0.f, 0.f, 0.f};
#pragma unroll
      for (int m = 0; m < 4; ++m) acc1[b][m][n] = (f32x4){0.f, 0.f, 0.f, 0.f};
    }
  i32x8 At[5], B0[2], B1[2];

  const char* fpa = myws + MBOX_OFF + (size_t)((tr * 16 + tj) * 128 + (lane < 3 ? lane : 0) * 8);
  u32x2 pa = {0u, 0u};
  unsigned done = 0xFu << (4 * rank), bad = 0;
  auto issue_poll = [&]() {
    asm volatile("global_load_dwordx2 %0, %1, off sc0 sc1" : "=&v"(pa) : "v"(fpa) : "memory");
  };
  auto ready_mask = [&]() -> unsigned {
    const unsigned mk = pa[0], tg = pa[1];
    const unsigned mv = (tg == call) ? mk : 0u;
    const unsigned m = (unsigned)__builtin_amdgcn_readlane(mv, 0) & (unsigned)__builtin_amdgcn_readlane(mv, 1) & (unsigned)__builtin_amdgcn_readlane(mv, 2);
    return m & ~done;
  };
  auto decide = [&](int slot) {
    asm volatile("" : "+v"(pa));
    unsigned m = ready_mask();
    if (m == 0u) {
      const unsigned long long t0 = __builtin_amdgcn_s_memrealtime();
      while (m == 0u) {
        if (bad == 0u) {
          issue_poll();
          asm volatile("s_waitcnt vmcnt(0)" : "+v"(pa) :: "memory");
          m = ready_mask();
          if (m == 0u) {
            const unsigned long long dt = __builtin_amdgcn_s_memrealtime() - t0;
            if (dt > 100000ull) {
              const unsigned tv = (unsigned)__builtin_amdgcn_readfirstlane(__hip_atomic_load(tmo, __ATOMIC_RELAXED, __HIP_MEMORY_SCOPE_AGENT));
              if (tv == call || dt > DEADLINE_TICKS) {
                if (lane == 0) { __hip_atomic_store(tmo, call, __ATOMIC_RELAXED, __HIP_MEMORY_SCOPE_AGENT); ctrl[2] = 1u; }
                bad = 1u;
              }
              __builtin_amdgcn_s_sleep(64);
            }
            __builtin_amdgcn_s_sleep(2);
          }
        } else {
          m = ~done;
        }
        m = (unsigned)__builtin_amdgcn_readfirstlane(m);
        bad = (unsigned)__builtin_amdgcn_readfirstlane(bad);
      }
    }
    const unsigned ks = (unsigned)__builtin_ctz(m);
    done = (unsigned)__builtin_amdgcn_readfirstlane(done | (1u << ks));
    if (lane == 0) ctrl[slot] = ks;
    issue_poll();
  };
#define READ_D(slot) ((unsigned)__builtin_amdgcn_readfirstlane(ctrl[slot]) * (512u * 1024u))

  const float scale = __builtin_bit_cast(float, __builtin_amdgcn_readfirstlane(__builtin_bit_cast(int, p.sx[0] * p.sw[0])));
  if (wid == 0) { if (lane == 0) ctrl[2] = 0u; issue_poll(); }
  STAGE_B(1, 0, 0, 0u); STAGE_A0(1, 0, 0u); STAGE_B(1, 0, 1, 0u); STAGE_A1(1, 0, 0u);
  if (wr == 1) BAR;
  WAIT_V(4); BAR;
  STAGE_B(1, 1, 0, 1u); STAGE_A0(1, 1, 1u); STAGE_B(1, 1, 1, 1u);
  WAIT_TILE; BAR;

  LDB(1, B0, 0, 0); LDA04(0);
#define TRIP(RL, S1L, SNL, more) do { \
    unsigned kb2 = 2u, kb3 = 3u; \
      \
    if (!(SNL) && wid == 0 && (more)) decide(0); \
    if (wid == 0 && !(more) && lane == 0) __builtin_amdgcn_raw_ptr_buffer_load_lds(tmoR, (LAS void*)(lds + L_CTRL + 12), 4, 0u, 0u, 0, 16);     \
    LDA0(0); STAGE_A1(S1L, 1, kb1); \
    BAR; WAIT_L(0); MMA0(0, B0); BAR; SCHED; \
      \
    if (!(SNL) && (more)) kb2 = READ_D(0); \
    LDB(RL, B1, 0, 1); if (more) STAGE_B(SNL, 0, 0, kb2); \
    BAR; WAIT_L(0); MMA0(1, B1); BAR; \
      \
    LDA1(0); if (more) STAGE_A0(SNL, 0, kb2); \
    WAIT_HALF(more); BAR; WAIT_L(0); MMA1(0, B0); BAR; SCHED; \
      \
    LDB(RL, B0, 1, 0); LDA04(1); \
    if (more) { STAGE_B(SNL, 0, 1, kb2); WAIT_TILE; } else { WAIT_V(0); } \
    BAR; MMA1(1, B1); BAR; \
      \
    if (!(SNL) && wid == 0 && (more)) decide(1); \
    LDA0(1); if (more) STAGE_A1(SNL, 0, kb2); \
    BAR; WAIT_L(0); MMA0(0, B0); BAR; SCHED; \
      \
    if (!(SNL) && (more)) kb3 = READ_D(1); \
    LDB(RL, B1, 1, 1); if (more) STAGE_B(SNL, 1, 0, kb3); \
    BAR; WAIT_L(0); MMA0(1, B1); BAR; \
      \
    LDA1(1); if (more) { STAGE_A0(SNL, 1, kb3); WAIT_HALF(true); } \
    BAR; WAIT_L(0); MMA1(0, B0); BAR; SCHED; \
      \
    if (more) { LDB(SNL, B0, 0, 0); LDA04(0); STAGE_B(SNL, 1, 1, kb3); WAIT_TILE; } \
    BAR; MMA1(1, B1); BAR; \
    kb1 = kb3; } while (0)
  unsigned kb1 = 1u;
#pragma unroll 1
  for (int gi = 0; gi < 2; ++gi) { const bool snl = gi == 0; TRIP(1, 1, snl, true); }
#pragma unroll 1
  for (int it = 2; it < 15; ++it) TRIP(0, 0, 0, true);
  { int once = 1; asm volatile("" : "+s"(once));
#pragma unroll 1
    for (int z = 0; z < once; ++z) TRIP(0, 0, 0, false); }
  if (wr == 0) BAR;

  const float escale = (ctrl[2] != 0u || ctrl[3] == call) ? __builtin_nanf("") : scale;
  int el = lane; asm volatile("" : "+v"(el));
  const int fr = el & 15, fq = el >> 4;
  LAS unsigned char* ewr = lds + wid * 8704 + fr * 272 + fq * 16;
  LAS unsigned char* erd = lds + wid * 8704 + fq * 272 + fr * 16;
  float* obase = p.out + (size_t)fq * 4096 + tj * 256 + wc * 64 + fr * 4;
  auto pass = [&](int j, f32x4& a00, f32x4& a01, f32x4& a10, f32x4& a11, int unit, bool live) {
    if (!live) return;
    const f32x4* src[4] = {&a00, &a01, &a10, &a11};
#pragma unroll
    for (int q = 0; q < 4; ++q) {
      f32x4 y = *src[q] * escale;
#pragma unroll
      for (int e = 0; e < 4; ++e) y[e] = (y[e] < 0.f) ? 0.f : y[e];
      *(LAS f32x4*)(ewr + (j & 1) * 4352 + q * 64) = y;
    }
    f32x4 r[4];
#pragma unroll
    for (int i = 0; i < 4; ++i) r[i] = *(const LAS f32x4*)(erd + (j & 1) * 4352 + i * 4 * 272);
#pragma unroll
    for (int i = 0; i < 4; ++i) asm volatile("global_store_dwordx4 %0, %1, off sc1\n\ts_nop 1" :: "v"(obase + (size_t)(unit * 16 + i * 4) * 4096), "v"(r[i]) : "memory");
  };
#pragma unroll
  for (int m = 0; m < 5; ++m) {
    const int slot = wr * 5 + m;
    pass(m, acc0[0][m][0], acc0[0][m][1], acc0[1][m][0], acc0[1][m][1], (slot < 9) ? U + slot : 255, slot < 9 || tr == 14);
  }
#pragma unroll
  for (int m = 0; m < 4; ++m) pass(5 + m, acc1[0][m][0], acc1[0][m][1], acc1[1][m][0], acc1[1][m][1], U + 9 + wr * 4 + m, true);
}
}

__global__ void __launch_bounds__(512, 2) fused_gemm_ag_kernel(Params p) {
  extern __shared__ __attribute__((aligned(16))) unsigned char smem_dyn[];
  LAS unsigned char* lds = (LAS unsigned char*)smem_dyn;
  const int wid = __builtin_amdgcn_readfirstlane(threadIdx.x >> 6), lane = threadIdx.x & 63;
  unsigned* ctr = (unsigned*)(p.ws[p.rank & 7] + CTR_OFF) + blockIdx.x * 16;
  unsigned call;
  if (blockIdx.x < NCOMM) comm_role(p, lds, ctr, call, wid, lane);
  else { call = (unsigned)__builtin_amdgcn_readfirstlane(ld_sys(ctr)) + 1u;
    gemm_role(p, lds, call, wid, lane);
  }
  if (threadIdx.x == 0) __hip_atomic_store(ctr, call, __ATOMIC_RELAXED, __HIP_MEMORY_SCOPE_SYSTEM);
}

extern "C" void kernel_launch(int rank, int nranks, ncclComm_t comm,
                              hipStream_t stream,
                              void* const* d_in, const int* in_sizes, int n_in,
                              void* d_out, int out_size,
                              const int* in_dtypes, int out_dtype,
                              void* const* d_out_all,
                              void* const* d_ws_all, size_t ws_bytes) {
  (void)comm; (void)in_sizes; (void)n_in; (void)out_size; (void)in_dtypes; (void)out_dtype; (void)d_out_all;
  if (nranks != NRANK || (unsigned)rank >= (unsigned)NRANK || ws_bytes < (size_t)WS_USED) {
    fprintf(stderr, "unsupported configuration\n"); return;
  }
  HIP_CHECK(hipFuncSetAttribute((const void*)fused_gemm_ag_kernel, hipFuncAttributeMaxDynamicSharedMemorySize, (int)LDS_BYTES));
  Params p;
  p.x = (const unsigned char*)d_in[0]; p.w = (const unsigned char*)d_in[1];
  p.sx = (const float*)d_in[2]; p.sw = (const float*)d_in[3];
  p.out = (float*)d_out; p.rank = rank;
  for (int i = 0; i < NRANK; ++i) p.ws[i] = (char*)d_ws_all[i];
  hipLaunchKernelGGL(fused_gemm_ag_kernel, dim3(NWG), dim3(512), LDS_BYTES, stream, p);
  HIP_CHECK(hipGetLastError());
}
```

```cpp
#include <hip/hip_runtime.h>
#include <rccl/rccl.h>
#include <cstdio>
#include <cstdint>

#define HIP_CHECK(expr) do { hipError_t _e = (expr); if (_e != hipSuccess) { \
    fprintf(stderr, "HIP error %s at %s:%d\n", hipGetErrorString(_e), __FILE__, __LINE__); \
    return; } } while (0)

#define LAS __attribute__((address_space(3)))
typedef int i32x8 __attribute__((ext_vector_type(8)));
typedef int i32x4 __attribute__((ext_vector_type(4)));
typedef int i32x2 __attribute__((ext_vector_type(2)));
typedef unsigned u32x4 __attribute__((ext_vector_type(4)));
typedef unsigned u32x2 __attribute__((ext_vector_type(2)));
typedef float f32x4 __attribute__((ext_vector_type(4)));
typedef unsigned short u16x2 __attribute__((ext_vector_type(2)));

namespace {
constexpr int NRANK = 8;
constexpr int NCOMM = 16, NGEMM = 240, NWG = NCOMM + NGEMM;
constexpr unsigned OPER_BYTES = 16u << 20;
constexpr unsigned BUF_BYTES = 2u * OPER_BYTES;
constexpr unsigned FLAG_OFF = 2u * BUF_BYTES;
constexpr unsigned FLAG_BYTES = 32u * 32u * 4u * 2u;
constexpr unsigned CTR_OFF = FLAG_OFF + (64u << 10);
constexpr unsigned TMO_OFF = FLAG_OFF + (128u << 10);
constexpr unsigned TICK_OFF = FLAG_OFF + (160u << 10);
constexpr unsigned MBOX_OFF = FLAG_OFF + (192u << 10);
constexpr unsigned WS_USED = FLAG_OFF + (256u << 10);
constexpr unsigned SHARD_BYTES = 2u << 20;
constexpr int TICK_Q = 4;
constexpr unsigned long long DEADLINE_TICKS = 50000000ull;

constexpr unsigned L_A0 = 0, L_A0B = 20480;
constexpr unsigned L_A1 = 40960, L_A1B = 16384;
constexpr unsigned L_B = 73728, L_BB = 32768;
constexpr unsigned L_CTRL = 139264;
constexpr unsigned C_RING = 1024;
constexpr unsigned LDS_BYTES = L_CTRL + 64;

struct Params {
  const unsigned char* x; const unsigned char* w; const float* sx; const float* sw; float* out;
  char* ws[NRANK]; int rank;
};

__device__ __forceinline__ __amdgpu_buffer_rsrc_t mk_rsrc(const void* p, unsigned bytes) {
  return __builtin_amdgcn_make_buffer_rsrc((void*)p, 0, bytes, 0x00020000);
}
__device__ __forceinline__ unsigned ld_sys(const unsigned* p) {
  return __hip_atomic_load(p, __ATOMIC_RELAXED, __HIP_MEMORY_SCOPE_SYSTEM);
}
#define CFENCE asm volatile("" ::: "memory")
#define WAIT_V(n) asm volatile("s_waitcnt vmcnt(" #n ")" ::: "memory")
#define WAIT_L(n) asm volatile("s_waitcnt lgkmcnt(" #n ")" ::: "memory")
#define BAR do { CFENCE; __builtin_amdgcn_s_barrier(); CFENCE; } while (0)
#define SCHED __builtin_amdgcn_sched_barrier(0)

__device__ __forceinline__ void comm_role(const Params& p, LAS unsigned char* lds, const unsigned* ctr, unsigned& call_out, int wid, int lane) {
  const int c = blockIdx.x;
  const int rank = p.rank & 7;
  LAS unsigned* cnt = (LAS unsigned*)lds;
  volatile LAS unsigned* ctl = (volatile LAS unsigned*)lds;
  if (threadIdx.x < 18) ctl[threadIdx.x] = (threadIdx.x >= 8 && threadIdx.x < 8 + TICK_Q) ? (unsigned)(c + 16 * (threadIdx.x - 8)) : (threadIdx.x == 16 ? (unsigned)TICK_Q : 0u);
  if (wid >= 4) {
    const unsigned call = (unsigned)__builtin_amdgcn_readfirstlane(ld_sys(ctr)) + 1u;
    call_out = call;
    WAIT_L(0); BAR;
    if (wid == 7) return;
    if (wid == 6) {
      unsigned* tick = (unsigned*)(p.ws[rank] + TICK_OFF) + (call & 1u) * 64;
      if (c == 0 && lane == 0) __hip_atomic_store((unsigned*)(p.ws[rank] + TICK_OFF) + ((call & 1u) ^ 1u) * 64, 0u, __ATOMIC_RELAXED, __HIP_MEMORY_SCOPE_AGENT);
      unsigned posted = TICK_Q;
      for (;;) {
        while (posted - (unsigned)__builtin_amdgcn_readfirstlane(ctl[17]) >= (unsigned)TICK_Q) __builtin_amdgcn_s_sleep(1);
        unsigned t = 0;
        if (lane == 0) t = __hip_atomic_fetch_add(tick, 1u, __ATOMIC_RELAXED, __HIP_MEMORY_SCOPE_AGENT);
        t = (unsigned)__builtin_amdgcn_readfirstlane(t) + 16u * TICK_Q;
        if (lane == 0) { ctl[8 + (posted & 7u)] = t; ctl[16] = posted + 1u; }
        ++posted;
        if (t >= 512u) break;
      }
      return;
    }
    const int e = 2 * c + (wid - 4);
    char* myws = p.ws[rank];
    const char* fp = myws + FLAG_OFF + (size_t)((e * 32 + (lane & 31)) * 8);
    int ctr_r, ctj, slot; bool tgt;
    if (e < 16) { ctr_r = (lane < 16) ? e : e - 1; ctj = lane & 15; slot = (lane < 16) ? 0 : 1; tgt = lane < 32 && (unsigned)ctr_r < 15u; }
    else { ctr_r = lane; ctj = e - 16; slot = 2; tgt = lane < 15; }
    unsigned long long* mb = (unsigned long long*)(myws + MBOX_OFF + (size_t)(((tgt ? ctr_r : 0) * 16 + ctj) * 128 + slot * 8));
    unsigned* tmo = (unsigned*)(myws + TMO_OFF);
    const unsigned short c16 = (unsigned short)call; const u16x2 cc = {c16, c16};
    unsigned last = 0;
    const unsigned long long t0 = __builtin_amdgcn_s_memrealtime();
    for (;;) {
      const unsigned long long fv = __hip_atomic_load((const unsigned long long*)fp, __ATOMIC_RELAXED, __HIP_MEMORY_SCOPE_SYSTEM);
      u16x2 a = __builtin_bit_cast(u16x2, (unsigned)fv), b = __builtin_bit_cast(u16x2, (unsigned)(fv >> 32));
      a -= cc; b -= cc;
      const unsigned dd = __builtin_bit_cast(unsigned, a) | __builtin_bit_cast(unsigned, b);
      const unsigned m = (unsigned)__builtin_amdgcn_ballot_w64((dd & 0xFFFEFFFEu) == 0u || ((lane & 31) >> 2) == rank);
      if (m != last) {
        if (tgt) __hip_atomic_store(mb, ((unsigned long long)call << 32) | m, __ATOMIC_RELAXED, __HIP_MEMORY_SCOPE_SYSTEM);
        last = m;
        if (m == 0xFFFFFFFFu) break;
      }
      if (__builtin_amdgcn_s_memrealtime() - t0 > DEADLINE_TICKS) {
        if (lane == 0) __hip_atomic_store(tmo, call, __ATOMIC_RELAXED, __HIP_MEMORY_SCOPE_AGENT);
        break;
      }
      __builtin_amdgcn_s_sleep(1);
    }
    return;
  }
  LAS unsigned char* ring = lds + C_RING + wid * 8192;
  const __amdgpu_buffer_rsrc_t xR = mk_rsrc(p.x, SHARD_BYTES), wR = mk_rsrc(p.w, SHARD_BYTES);
  const unsigned lp_w = (unsigned)((lane >> 2) * 4096 + (lane & 3) * 16), lp_x = (unsigned)((lane & 15) * 512 + (lane >> 4) * 16);
  auto blk = [&](unsigned t, int& o, int& eb, int& s, int& q4) {
    const int idx = (int)((t < 512u ? t : 511u) >> 2);
    q4 = (int)(t & 3u); o = idx & 1; s = idx >> 5;
    eb = (((idx >> 1) & 15) + 2 * rank) & 15;
  };
  auto fetch = [&](unsigned t, unsigned slot) {
    int o, eb, s, q4; blk(t, o, eb, s, q4);
    const int u0 = eb * 16 + 4 * q4;
    LAS unsigned char* dst = ring + (slot & 3u) * 2048;
    if (o) {
      const unsigned so = (unsigned)((s * 128 + 32 * wid) * 4096 + u0 * 16);
      __builtin_amdgcn_raw_ptr_buffer_load_lds(wR, (LAS void*)dst, 16, lp_w, so, 0, 0);
      __builtin_amdgcn_raw_ptr_buffer_load_lds(wR, (LAS void*)(dst + 1024), 16, lp_w, so + 16u * 4096u, 0, 0);
    } else {
      const unsigned so = (unsigned)((u0 + wid) * 8192 + s * 128);
      __builtin_amdgcn_raw_ptr_buffer_load_lds(xR, (LAS void*)dst, 16, lp_x, so, 0, 0);
      __builtin_amdgcn_raw_ptr_buffer_load_lds(xR, (LAS void*)(dst + 1024), 16, lp_x, so + 64u, 0, 0);
    }
  };
  const unsigned craw = ld_sys(ctr);
  unsigned t0 = (unsigned)c, t1 = (unsigned)c + 16u, tprev = 0;
  fetch(t0, 0); fetch(t1, 1);
  const unsigned call = (unsigned)__builtin_amdgcn_readfirstlane(craw) + 1u;
  asm volatile("" :: "s"(call));
  call_out = call;
  __amdgpu_buffer_rsrc_t dstR[7];
  {
    const int rot = (c * 4 + wid) % 7;
#pragma unroll
    for (int q = 0; q < 7; ++q) { const int pr = (rank + 1 + (q + rot) % 7) & 7; dstR[q] = mk_rsrc(p.ws[pr], FLAG_OFF); }
  }
  unsigned flo = 0, fhi = 0;
#pragma unroll
  for (int q = 0; q < 8; ++q) {
    const unsigned long long a = (unsigned long long)(p.ws[q] + FLAG_OFF);
    unsigned lo = (unsigned)a, hi = (unsigned)(a >> 32);
    asm volatile("" : "+s"(lo), "+s"(hi));
    flo = ((lane & 7) == q) ? lo : flo; fhi = ((lane & 7) == q) ? hi : fhi;
  }
  typedef __attribute__((address_space(1))) unsigned short gu16;
  gu16* flp = (gu16*)(((unsigned long long)fhi << 32) | flo);
  const unsigned vo_w = (unsigned)((lane >> 5) * 2048 + ((lane >> 4) & 1) * 256 + (lane & 15) * 16), vo_x = (unsigned)lane * 16u;
  const unsigned dcommon = (call & 1u) * BUF_BYTES;
  const unsigned db_w = dcommon + OPER_BYTES + (unsigned)((wid >> 1) * 1024 + (wid & 1) * 512), db_x = dcommon + (unsigned)wid * 2048u;
  const unsigned rd_w = (unsigned)(size_t)ring + (unsigned)((((lane >> 4) & 1) * 16 + ((lane & 15) >> 1)) * 64 + (lane >> 5) * 16 + (lane & 1) * 8);
  const unsigned rd_x = (unsigned)(size_t)ring + (unsigned)lane * 16u;
  WAIT_L(0); BAR;

  auto signal = [&](unsigned i, unsigned t) {
    unsigned old = 0;
    if (lane == 0) old = __hip_atomic_fetch_add(cnt + (i & 7u), 1u, __ATOMIC_RELAXED, __HIP_MEMORY_SCOPE_WORKGROUP);
    old = __builtin_amdgcn_readfirstlane(old);
    if (old == 3u) {
      int o, eb, s, q4; blk(t, o, eb, s, q4);
      const unsigned idx = (unsigned)((((o * 16 + eb) * 32) + rank * 4 + s) * 4 + q4);
      if (lane < 8 && lane != rank) __hip_atomic_store(flp + idx, (unsigned short)call, __ATOMIC_RELAXED, __HIP_MEMORY_SCOPE_SYSTEM);
      if (lane == 0) { ctl[i & 7u] = 0u; __hip_atomic_fetch_add(cnt + 17, 1u, __ATOMIC_RELAXED, __HIP_MEMORY_SCOPE_WORKGROUP); }
    }
  };
  unsigned i = 0;
#pragma unroll 1
  for (; t0 < 512u; ++i) {
    if (i == 0) WAIT_V(2); else WAIT_V(16);
    int o, eb, s, q4; blk(t0, o, eb, s, q4);
    u32x4 v0, v1;
    if (o) {
      const unsigned a = rd_w + (i & 3u) * 2048u;
      u32x2 a0, a1, b0, b1;
      asm volatile("ds_read_b64_tr_b8 %0, %4\n\tds_read_b64_tr_b8 %1, %4 offset:512\n\t"
                   "ds_read_b64_tr_b8 %2, %4 offset:32\n\tds_read_b64_tr_b8 %3, %4 offset:544\n\ts_waitcnt lgkmcnt(0)"
                   : "=&v"(a0), "=&v"(a1), "=&v"(b0), "=&v"(b1) : "v"(a) : "memory");
      v0 = (u32x4){a0.x, a0.y, a1.x, a1.y}; v1 = (u32x4){b0.x, b0.y, b1.x, b1.y};
    } else {
      const unsigned a = rd_x + (i & 3u) * 2048u;
      asm volatile("ds_read_b128 %0, %2\n\tds_read_b128 %1, %2 offset:1024\n\ts_waitcnt lgkmcnt(0)"
                   : "=&v"(v0), "=&v"(v1) : "v"(a) : "memory");
    }
    const unsigned voff = o ? vo_w : vo_x;
    const unsigned doff = (o ? db_w : db_x) + (unsigned)(((rank * 4 + s) * 256 + eb * 16 + 4 * q4) * 2048);
    const unsigned doff1 = doff + (o ? 4096u : 1024u);
#pragma unroll
    for (int q = 0; q < 7; ++q) {
      __builtin_amdgcn_raw_buffer_store_b128(v0, dstR[q], voff, doff, 17);
      __builtin_amdgcn_raw_buffer_store_b128(v1, dstR[q], voff, doff1, 17);
    }
    if (i >= 1) {
      WAIT_V(16);
      signal(i - 1, tprev);
    }
    unsigned t2 = 512u;
    if (t1 < 512u) {
      while ((unsigned)__builtin_amdgcn_readfirstlane(ctl[16]) <= i + 2u) __builtin_amdgcn_s_sleep(1);
      t2 = (unsigned)__builtin_amdgcn_readfirstlane(ctl[8 + ((i + 2u) & 7u)]);
    }
    fetch(t2, i + 2u);
    tprev = t0; t0 = t1; t1 = t2;
  }
  WAIT_V(0); signal(i - 1, tprev);
}

__device__ __forceinline__ i32x8 ld_frag(LAS unsigned char* a) {
  const i32x4 lo = *(const LAS i32x4*)a, hi = *(const LAS i32x4*)(a + 1024);
  return __builtin_shufflevector(lo, hi, 0, 1, 2, 3, 4, 5, 6, 7);
}

__device__ __forceinline__ i32x8 ld_frag_tr(LAS unsigned char* a) {
  const i32x2 q0 = __builtin_amdgcn_ds_read_tr8_b64_v2i32((LAS i32x2*)a), q1 = __builtin_amdgcn_ds_read_tr8_b64_v2i32((LAS i32x2*)(a + 1024));
  const i32x2 q2 = __builtin_amdgcn_ds_read_tr8_b64_v2i32((LAS i32x2*)(a + 8192)), q3 = __builtin_amdgcn_ds_read_tr8_b64_v2i32((LAS i32x2*)(a + 9216));
  return (i32x8){q0.x, q0.y, q1.x, q1.y, q2.x, q2.y, q3.x, q3.y};
}

__device__ __forceinline__ void gemm_role(const Params& p, LAS unsigned char* lds, unsigned call, int wid, int lane) {
  const int g = blockIdx.x - NCOMM;
  const int L = 30 * (g & 7) + (g >> 3);
  const int tj = (L < 192) ? ((L & 63) >> 2) : (L - 192) / 3, tr = (L < 192) ? ((L >> 6) * 4 + (L & 3)) : 12 + (L - 192) % 3;
  const int wr = wid >> 2, wc = wid & 3;
  const int rank = p.rank & 7;
  char* myws = p.ws[rank];
  const int U = 17 * tr, UX = (tr == 14) ? 255 : U + 8;
  const __amdgpu_buffer_rsrc_t opR = mk_rsrc(myws + (size_t)(call & 1u) * BUF_BYTES, BUF_BYTES);
  const __amdgpu_buffer_rsrc_t xR = mk_rsrc(p.x, SHARD_BYTES), wR = mk_rsrc(p.w, SHARD_BYTES);
  const unsigned voff = (unsigned)lane * 16u;
  volatile LAS unsigned* ctrl = (volatile LAS unsigned*)(lds + L_CTRL);
  unsigned* tmo = (unsigned*)(myws + TMO_OFF);
  const __amdgpu_buffer_rsrc_t tmoR = mk_rsrc(tmo, 64);

  const unsigned sA0a = (unsigned)(U * 2 + wid) * 1024u;
  const unsigned sA0x = (wid < 6) ? (unsigned)(U * 2 + 12 + wid) * 1024u : (unsigned)(UX * 2 + wid - 6) * 1024u;
  const unsigned sA1 = (unsigned)((U + 9) * 2 + wid) * 1024u;
  const unsigned sB = OPER_BYTES + (unsigned)((tj * 16 + 4 * (wid >> 2) + ((wid >> 1) & 1)) * 2048 + (wid & 1) * 1024);
  const unsigned vx = (unsigned)((lane & 15) * 512 + (lane >> 4) * 16);
  const int vch = (lane & 7) ^ (((lane >> 4) & 3) | ((wid & 2) << 1));
  const unsigned vw = (unsigned)((lane >> 3) * 4096 + (vch >> 1) * 64 + (vch & 1) * 16);
  const unsigned lA0a = (unsigned)((U + (wid >> 1)) * 8192 + (wid & 1) * 64);
  const unsigned lA0x = (unsigned)(((wid < 6) ? U + 8 : UX) * 8192 + (wid & 1) * 64);
  const unsigned lA1 = lA0a + 9u * 8192u;
  const unsigned lB = (unsigned)(wid * 8 * 4096 + tj * 256);
  const unsigned lw = (unsigned)wid * 1024u;
#define GLDS(ldsoff, soff) __builtin_amdgcn_raw_ptr_buffer_load_lds(opR, (LAS void*)(lds + (ldsoff)), 16, voff, (soff), 0, 17)
#define GLDX(ldsoff, soff) __builtin_amdgcn_raw_ptr_buffer_load_lds(xR, (LAS void*)(lds + (ldsoff)), 16, vx, (soff), 0, 0)
#define GLDW(ldsoff, soff) __builtin_amdgcn_raw_ptr_buffer_load_lds(wR, (LAS void*)(lds + (ldsoff)), 16, vw, (soff), 0, 0)
#define STAGE_A0(LOC, b, kb) do { if (LOC) { GLDX(L_A0 + (b) * L_A0B + lw, lA0a + (kb) * 128u); GLDX(L_A0 + (b) * L_A0B + lw + 8192, lA0a + 32768u + (kb) * 128u); \
                                             if (wr) GLDX(L_A0 + (b) * L_A0B + lw + 12288, lA0x + (kb) * 128u); } \
                                  else { GLDS(L_A0 + (b) * L_A0B + lw, (kb) + sA0a); GLDS(L_A0 + (b) * L_A0B + lw + 8192, (kb) + sA0a + 8192); \
                                         if (wr) GLDS(L_A0 + (b) * L_A0B + lw + 12288, (kb) + sA0x); } } while (0)
#define STAGE_A1(LOC, b, kb) do { if (LOC) { GLDX(L_A1 + (b) * L_A1B + lw, lA1 + (kb) * 128u); GLDX(L_A1 + (b) * L_A1B + lw + 8192, lA1 + 32768u + (kb) * 128u); } \
                                  else { GLDS(L_A1 + (b) * L_A1B + lw, (kb) + sA1); GLDS(L_A1 + (b) * L_A1B + lw + 8192, (kb) + sA1 + 8192); } } while (0)
#define STAGE_B(LOC, b, bj, kb) do { if (LOC) { GLDW(L_B + (b) * L_BB + (bj) * 16384 + lw, lB + (kb) * 524288u + (bj) * 32); \
                                                GLDW(L_B + (b) * L_BB + (bj) * 16384 + lw + 8192, lB + (kb) * 524288u + (bj) * 32 + 262144u); } \
                                     else { GLDS(L_B + (b) * L_BB + (bj) * 16384 + lw, (kb) + sB + (bj) * 4096); \
                                            GLDS(L_B + (b) * L_BB + (bj) * 16384 + lw + 8192, (kb) + sB + (bj) * 4096 + 16384); } } while (0)
#define WAIT_TILE do { if (wr) WAIT_V(7); else WAIT_V(6); } while (0)

  const bool use5 = (wr == 0) || (tr == 14);
  LAS unsigned char* rA0 = lds + L_A0 + wr * 10240 + lane * 16;
  LAS unsigned char* rA1 = lds + L_A1 + wr * 8192 + lane * 16;
  LAS unsigned char* rB = lds + L_B + wc * 4096 + lane * 16;
  const int fsw = (((lane & 15) >> 2) & 3) | (((lane >> 4) & 1) << 2);
  LAS unsigned char* rBl0 = lds + L_B + ((lane >> 4) * 16 + ((lane & 15) >> 1)) * 128 + (((wc * 2) ^ fsw) * 16) + (lane & 1) * 8;
  LAS unsigned char* rBl1 = lds + L_B + ((lane >> 4) * 16 + ((lane & 15) >> 1)) * 128 + (((wc * 2 + 1) ^ fsw) * 16) + (lane & 1) * 8;
#define LDA0(b) do { _Pragma("unroll") for (int m = 0; m < 4; ++m) At[m] = ld_frag(rA0 + (b) * L_A0B + m * 2048); } while (0)
#define LDA04(b) do { At[4] = ld_frag(rA0 + (b) * L_A0B + 4 * 2048); } while (0)
#define WAIT_HALF(more) do { if (more) { if (wr) WAIT_V(9); else WAIT_V(8); } else WAIT_V(4); } while (0)
#define LDA1(b) do { _Pragma("unroll") for (int m = 0; m < 4; ++m) At[m] = ld_frag(rA1 + (b) * L_A1B + m * 2048); } while (0)
#define LDB(LOC, dst, b, bj) do { if (LOC) { dst[0] = ld_frag_tr(rBl0 + (b) * L_BB + (bj) * 16384); dst[1] = ld_frag_tr(rBl1 + (b) * L_BB + (bj) * 16384); } \
    else { _Pragma("unroll") for (int n = 0; n < 2; ++n) dst[n] = ld_frag(rB + (b) * L_BB + (bj) * 16384 + n * 2048); } } while (0)
#define MMA0(bj, Bt) do { __builtin_amdgcn_s_setprio(1); _Pragma("unroll") for (int m = 0; m < 4; ++m) _Pragma("unroll") for (int n = 0; n < 2; ++n) \
    acc0[bj][m][n] = __builtin_amdgcn_mfma_scale_f32_16x16x128_f8f6f4(Bt[n], At[m], acc0[bj][m][n], 0, 0, 0, 0, 0, 0); \
    if (use5) { _Pragma("unroll") for (int n = 0; n < 2; ++n) \
    acc0[bj][4][n] = __builtin_amdgcn_mfma_scale_f32_16x16x128_f8f6f4(Bt[n], At[4], acc0[bj][4][n], 0, 0, 0, 0, 0, 0); } __builtin_amdgcn_s_setprio(0); } while (0)
#define MMA1(bj, Bt) do { __builtin_amdgcn_s_setprio(1); _Pragma("unroll") for (int m = 0; m < 4; ++m) _Pragma("unroll") for (int n = 0; n < 2; ++n) \
    acc1[bj][m][n] = __builtin_amdgcn_mfma_scale_f32_16x16x128_f8f6f4(Bt[n], At[m], acc1[bj][m][n], 0, 0, 0, 0, 0, 0); __builtin_amdgcn_s_setprio(0); } while (0)

  f32x4 acc0[2][5][2], acc1[2][4][2];
#pragma unroll
  for (int b = 0; b < 2; ++b)
#pragma unroll
    for (int n = 0; n < 2; ++n) {
#pragma unroll
      for (int m = 0; m < 5; ++m) acc0[b][m][n] = (f32x4){0.f, 0.f, 0.f, 0.f};
#pragma unroll
      for (int m = 0; m < 4; ++m) acc1[b][m][n] = (f32x4){0.f, 0.f, 0.f, 0.f};
    }
  i32x8 At[5], B0[2], B1[2];

  const char* fpa = myws + MBOX_OFF + (size_t)((tr * 16 + tj) * 128 + (lane < 3 ? lane : 0) * 8);
  u32x2 pa = {0u, 0u};
  unsigned done = 0xFu << (4 * rank), bad = 0;
  auto issue_poll = [&]() {
    asm volatile("global_load_dwordx2 %0, %1, off sc0 sc1" : "=&v"(pa) : "v"(fpa) : "memory");
  };
  auto ready_mask = [&]() -> unsigned {
    const unsigned mk = pa[0], tg = pa[1];
    const unsigned mv = (tg == call) ? mk : 0u;
    const unsigned m = (unsigned)__builtin_amdgcn_readlane(mv, 0) & (unsigned)__builtin_amdgcn_readlane(mv, 1) & (unsigned)__builtin_amdgcn_readlane(mv, 2);
    return m & ~done;
  };
  auto decide = [&](int slot) {
    asm volatile("" : "+v"(pa));
    unsigned m = ready_mask();
    if (m == 0u) {
      const unsigned long long t0 = __builtin_amdgcn_s_memrealtime();
      while (m == 0u) {
        if (bad == 0u) {
          issue_poll();
          asm volatile("s_waitcnt vmcnt(0)" : "+v"(pa) :: "memory");
          m = ready_mask();
          if (m == 0u) {
            const unsigned long long dt = __builtin_amdgcn_s_memrealtime() - t0;
            if (dt > 100000ull) {
              const unsigned tv = (unsigned)__builtin_amdgcn_readfirstlane(__hip_atomic_load(tmo, __ATOMIC_RELAXED, __HIP_MEMORY_SCOPE_AGENT));
              if (tv == call || dt > DEADLINE_TICKS) {
                if (lane == 0) { __hip_atomic_store(tmo, call, __ATOMIC_RELAXED, __HIP_MEMORY_SCOPE_AGENT); ctrl[2] = 1u; }
                bad = 1u;
              }
              __builtin_amdgcn_s_sleep(64);
            }
            __builtin_amdgcn_s_sleep(2);
          }
        } else {
          m = ~done;
        }
        m = (unsigned)__builtin_amdgcn_readfirstlane(m);
        bad = (unsigned)__builtin_amdgcn_readfirstlane(bad);
      }
    }
    const unsigned ks = (unsigned)__builtin_ctz(m);
    done = (unsigned)__builtin_amdgcn_readfirstlane(done | (1u << ks));
    if (lane == 0) ctrl[slot] = ks;
    issue_poll();
  };
#define READ_D(slot) ((unsigned)__builtin_amdgcn_readfirstlane(ctrl[slot]) * (512u * 1024u))

  const float scale = __builtin_bit_cast(float, __builtin_amdgcn_readfirstlane(__builtin_bit_cast(int, p.sx[0] * p.sw[0])));
  if (wid == 0) { if (lane == 0) ctrl[2] = 0u; issue_poll(); }
  STAGE_B(1, 0, 0, 0u); STAGE_A0(1, 0, 0u); STAGE_B(1, 0, 1, 0u); STAGE_A1(1, 0, 0u);
  if (wr == 1) BAR;
  WAIT_V(4); BAR;
  STAGE_B(1, 1, 0, 1u); STAGE_A0(1, 1, 1u); STAGE_B(1, 1, 1, 1u);
  WAIT_TILE; BAR;

  LDB(1, B0, 0, 0); LDA04(0);
#define TRIP(RL, S1L, SNL, more) do { \
    unsigned kb2 = 2u, kb3 = 3u; \
      \
    if (!(SNL) && wid == 0 && (more)) decide(0); \
    if (wid == 0 && !(more) && lane == 0) __builtin_amdgcn_raw_ptr_buffer_load_lds(tmoR, (LAS void*)(lds + L_CTRL + 12), 4, 0u, 0u, 0, 16);     \
    LDA0(0); STAGE_A1(S1L, 1, kb1); \
    BAR; WAIT_L(0); MMA0(0, B0); BAR; SCHED; \
      \
    if (!(SNL) && (more)) kb2 = READ_D(0); \
    LDB(RL, B1, 0, 1); if (more) STAGE_B(SNL, 0, 0, kb2); \
    BAR; WAIT_L(0); MMA0(1, B1); BAR; \
      \
    LDA1(0); if (more) STAGE_A0(SNL, 0, kb2); \
    WAIT_HALF(more); BAR; WAIT_L(0); MMA1(0, B0); BAR; SCHED; \
      \
    LDB(RL, B0, 1, 0); LDA04(1); \
    if (more) { STAGE_B(SNL, 0, 1, kb2); WAIT_TILE; } else { WAIT_V(0); } \
    BAR; MMA1(1, B1); BAR; \
      \
    if (!(SNL) && wid == 0 && (more)) decide(1); \
    LDA0(1); if (more) STAGE_A1(SNL, 0, kb2); \
    BAR; WAIT_L(0); MMA0(0, B0); BAR; SCHED; \
      \
    if (!(SNL) && (more)) kb3 = READ_D(1); \
    LDB(RL, B1, 1, 1); if (more) STAGE_B(SNL, 1, 0, kb3); \
    BAR; WAIT_L(0); MMA0(1, B1); BAR; \
      \
    LDA1(1); if (more) { STAGE_A0(SNL, 1, kb3); WAIT_HALF(true); } \
    BAR; WAIT_L(0); MMA1(0, B0); BAR; SCHED; \
      \
    if (more) { LDB(SNL, B0, 0, 0); LDA04(0); STAGE_B(SNL, 1, 1, kb3); WAIT_TILE; } \
    BAR; MMA1(1, B1); BAR; \
    kb1 = kb3; } while (0)
  unsigned kb1 = 1u;
#pragma unroll 1
  for (int gi = 0; gi < 2; ++gi) { const bool snl = gi == 0; TRIP(1, 1, snl, true); }
  {
    int pl = lane; asm volatile("" : "+v"(pl));
    float* pb = p.out + (size_t)(pl >> 4) * 4096 + tj * 256 + wc * 64 + (pl & 15) * 4;
    const f32x4 z = {0.f, 0.f, 0.f, 0.f};
#pragma unroll 1
    for (int j = 0; j < 9; ++j) {
      const int unit = (j < 5) ? U + wr * 5 + j : U + 9 + wr * 4 + (j - 5);
      if (j != 4 || wr == 0) {
#pragma unroll
        for (int i = 0; i < 4; ++i) asm volatile("global_store_dwordx4 %0, %1, off sc1\n\ts_nop 1" :: "v"(pb + (size_t)(unit * 16 + i * 4) * 4096), "v"(z) : "memory");
      }
      __builtin_amdgcn_s_sleep(32);
    }
  }
#pragma unroll 1
  for (int it = 2; it < 15; ++it) TRIP(0, 0, 0, true);
  { int once = 1; asm volatile("" : "+s"(once));
#pragma unroll 1
    for (int z = 0; z < once; ++z) TRIP(0, 0, 0, false); }
  if (wr == 0) BAR;

  const float escale = (ctrl[2] != 0u || ctrl[3] == call) ? __builtin_nanf("") : scale;
  int el = lane; asm volatile("" : "+v"(el));
  const int fr = el & 15, fq = el >> 4;
  LAS unsigned char* ewr = lds + wid * 8704 + fr * 272 + fq * 16;
  LAS unsigned char* erd = lds + wid * 8704 + fq * 272 + fr * 16;
  float* obase = p.out + (size_t)fq * 4096 + tj * 256 + wc * 64 + fr * 4;
  auto pass = [&](int j, f32x4& a00, f32x4& a01, f32x4& a10, f32x4& a11, int unit, bool live) {
    if (!live) return;
    const f32x4* src[4] = {&a00, &a01, &a10, &a11};
#pragma unroll
    for (int q = 0; q < 4; ++q) {
      f32x4 y = *src[q] * escale;
#pragma unroll
      for (int e = 0; e < 4; ++e) y[e] = (y[e] < 0.f) ? 0.f : y[e];
      *(LAS f32x4*)(ewr + (j & 1) * 4352 + q * 64) = y;
    }
    f32x4 r[4];
#pragma unroll
    for (int i = 0; i < 4; ++i) r[i] = *(const LAS f32x4*)(erd + (j & 1) * 4352 + i * 4 * 272);
#pragma unroll
    for (int i = 0; i < 4; ++i) asm volatile("global_store_dwordx4 %0, %1, off sc1\n\ts_nop 1" :: "v"(obase + (size_t)(unit * 16 + i * 4) * 4096), "v"(r[i]) : "memory");
  };
#pragma unroll
  for (int m = 0; m < 5; ++m) {
    const int slot = wr * 5 + m;
    pass(m, acc0[0][m][0], acc0[0][m][1], acc0[1][m][0], acc0[1][m][1], (slot < 9) ? U + slot : 255, slot < 9 || tr == 14);
  }
#pragma unroll
  for (int m = 0; m < 4; ++m) pass(5 + m, acc1[0][m][0], acc1[0][m][1], acc1[1][m][0], acc1[1][m][1], U + 9 + wr * 4 + m, true);
}
}

__global__ void __launch_bounds__(512, 2) fused_gemm_ag_kernel(Params p) {
  extern __shared__ __attribute__((aligned(16))) unsigned char smem_dyn[];
  LAS unsigned char* lds = (LAS unsigned char*)smem_dyn;
  const int wid = __builtin_amdgcn_readfirstlane(threadIdx.x >> 6), lane = threadIdx.x & 63;
  unsigned* ctr = (unsigned*)(p.ws[p.rank & 7] + CTR_OFF) + blockIdx.x * 16;
  unsigned call;
  if (blockIdx.x < NCOMM) comm_role(p, lds, ctr, call, wid, lane);
  else { call = (unsigned)__builtin_amdgcn_readfirstlane(ld_sys(ctr)) + 1u;
    gemm_role(p, lds, call, wid, lane);
  }
  if (threadIdx.x == 0) __hip_atomic_store(ctr, call, __ATOMIC_RELAXED, __HIP_MEMORY_SCOPE_SYSTEM);
}

extern "C" void kernel_launch(int rank, int nranks, ncclComm_t comm,
                              hipStream_t stream,
                              void* const* d_in, const int* in_sizes, int n_in,
                              void* d_out, int out_size,
                              const int* in_dtypes, int out_dtype,
                              void* const* d_out_all,
                              void* const* d_ws_all, size_t ws_bytes) {
  (void)comm; (void)in_sizes; (void)n_in; (void)out_size; (void)in_dtypes; (void)out_dtype; (void)d_out_all;
  if (nranks != NRANK || (unsigned)rank >= (unsigned)NRANK || ws_bytes < (size_t)WS_USED) {
    fprintf(stderr, "unsupported configuration\n"); return;
  }
  HIP_CHECK(hipFuncSetAttribute((const void*)fused_gemm_ag_kernel, hipFuncAttributeMaxDynamicSharedMemorySize, (int)LDS_BYTES));
  Params p;
  p.x = (const unsigned char*)d_in[0]; p.w = (const unsigned char*)d_in[1];
  p.sx = (const float*)d_in[2]; p.sw = (const float*)d_in[3];
  p.out = (float*)d_out; p.rank = rank;
  for (int i = 0; i < NRANK; ++i) p.ws[i] = (char*)d_ws_all[i];
  hipLaunchKernelGGL(fused_gemm_ag_kernel, dim3(NWG), dim3(512), LDS_BYTES, stream, p);
  HIP_CHECK(hipGetLastError());
}
```

```cpp
#include <hip/hip_runtime.h>
#include <rccl/rccl.h>
#include <cstdio>
#include <cstdint>

#define HIP_CHECK(expr) do { hipError_t _e = (expr); if (_e != hipSuccess) { \
    fprintf(stderr, "HIP error %s at %s:%d\n", hipGetErrorString(_e), __FILE__, __LINE__); \
    return; } } while (0)

#define LAS __attribute__((address_space(3)))
typedef int i32x8 __attribute__((ext_vector_type(8)));
typedef int i32x4 __attribute__((ext_vector_type(4)));
typedef int i32x2 __attribute__((ext_vector_type(2)));
typedef unsigned u32x4 __attribute__((ext_vector_type(4)));
typedef unsigned u32x2 __attribute__((ext_vector_type(2)));
typedef float f32x4 __attribute__((ext_vector_type(4)));
typedef unsigned short u16x2 __attribute__((ext_vector_type(2)));

namespace {
constexpr int NRANK = 8;
constexpr int NCOMM = 16, NGEMM = 240, NWG = NCOMM + NGEMM;
constexpr unsigned OPER_BYTES = 16u << 20;
constexpr unsigned BUF_BYTES = 2u * OPER_BYTES;
constexpr unsigned FLAG_OFF = 2u * BUF_BYTES;
constexpr unsigned FLAG_BYTES = 32u * 32u * 4u * 2u;
constexpr unsigned CTR_OFF = FLAG_OFF + (64u << 10);
constexpr unsigned TMO_OFF = FLAG_OFF + (128u << 10);
constexpr unsigned TICK_OFF = FLAG_OFF + (160u << 10);
constexpr unsigned WS_USED = FLAG_OFF + (256u << 10);
constexpr unsigned SHARD_BYTES = 2u << 20;
constexpr int TICK_Q = 4;
constexpr unsigned long long DEADLINE_TICKS = 50000000ull;

constexpr unsigned L_A0 = 0, L_A0B = 20480;
constexpr unsigned L_A1 = 40960, L_A1B = 16384;
constexpr unsigned L_B = 73728, L_BB = 32768;
constexpr unsigned L_CTRL = 139264;
constexpr unsigned C_RING = 1024;
constexpr unsigned LDS_BYTES = L_CTRL + 64;

struct Params {
  const unsigned char* x; const unsigned char* w; const float* sx; const float* sw; float* out;
  char* ws[NRANK]; int rank;
};

__device__ __forceinline__ __amdgpu_buffer_rsrc_t mk_rsrc(const void* p, unsigned bytes) {
  return __builtin_amdgcn_make_buffer_rsrc((void*)p, 0, bytes, 0x00020000);
}
__device__ __forceinline__ unsigned ld_sys(const unsigned* p) {
  return __hip_atomic_load(p, __ATOMIC_RELAXED, __HIP_MEMORY_SCOPE_SYSTEM);
}
#define CFENCE asm volatile("" ::: "memory")
#define WAIT_V(n) asm volatile("s_waitcnt vmcnt(" #n ")" ::: "memory")
#define WAIT_L(n) asm volatile("s_waitcnt lgkmcnt(" #n ")" ::: "memory")
#define BAR do { CFENCE; __builtin_amdgcn_s_barrier(); CFENCE; } while (0)
#define SCHED __builtin_amdgcn_sched_barrier(0)

__device__ __forceinline__ void comm_role(const Params& p, LAS unsigned char* lds, const unsigned* ctr, unsigned& call_out, int wid, int lane) {
  const int c = blockIdx.x;
  const int rank = p.rank & 7;
  LAS unsigned* cnt = (LAS unsigned*)lds;
  volatile LAS unsigned* ctl = (volatile LAS unsigned*)lds;
  if (threadIdx.x < 18) ctl[threadIdx.x] = (threadIdx.x >= 8 && threadIdx.x < 8 + TICK_Q) ? (unsigned)(c + 16 * (threadIdx.x - 8)) : (threadIdx.x == 16 ? (unsigned)TICK_Q : 0u);
  if (wid >= 4) {
    const unsigned call = (unsigned)__builtin_amdgcn_readfirstlane(ld_sys(ctr)) + 1u;
    call_out = call;
    WAIT_L(0); BAR;
    if (wid == 4) {
      unsigned* tick = (unsigned*)(p.ws[rank] + TICK_OFF) + (call & 1u) * 64;
      if (c == 0 && lane == 0) __hip_atomic_store((unsigned*)(p.ws[rank] + TICK_OFF) + ((call & 1u) ^ 1u) * 64, 0u, __ATOMIC_RELAXED, __HIP_MEMORY_SCOPE_AGENT);
      unsigned posted = TICK_Q;
      for (;;) {
        while (posted - (unsigned)__builtin_amdgcn_readfirstlane(ctl[17]) >= (unsigned)TICK_Q) __builtin_amdgcn_s_sleep(1);
        unsigned t = 0;
        if (lane == 0) t = __hip_atomic_fetch_add(tick, 1u, __ATOMIC_RELAXED, __HIP_MEMORY_SCOPE_AGENT);
        t = (unsigned)__builtin_amdgcn_readfirstlane(t) + 16u * TICK_Q;
        if (lane == 0) { ctl[8 + (posted & 7u)] = t; ctl[16] = posted + 1u; }
        ++posted;
        if (t >= 512u) break;
      }
      return;
    }
    return;
  }
  LAS unsigned char* ring = lds + C_RING + wid * 8192;
  const __amdgpu_buffer_rsrc_t xR = mk_rsrc(p.x, SHARD_BYTES), wR = mk_rsrc(p.w, SHARD_BYTES);
  const unsigned lp_w = (unsigned)((lane >> 2) * 4096 + (lane & 3) * 16), lp_x = (unsigned)((lane & 15) * 512 + (lane >> 4) * 16);
  auto blk = [&](unsigned t, int& o, int& eb, int& s, int& q4) {
    const int idx = (int)((t < 512u ? t : 511u) >> 2);
    q4 = (int)(t & 3u); o = idx & 1; s = idx >> 5;
    eb = (((idx >> 1) & 15) + 2 * rank) & 15;
  };
  auto fetch = [&](unsigned t, unsigned slot) {
    int o, eb, s, q4; blk(t, o, eb, s, q4);
    const int u0 = eb * 16 + 4 * q4;
    LAS unsigned char* dst = ring + (slot & 3u) * 2048;
    if (o) {
      const unsigned so = (unsigned)((s * 128 + 32 * wid) * 4096 + u0 * 16);
      __builtin_amdgcn_raw_ptr_buffer_load_lds(wR, (LAS void*)dst, 16, lp_w, so, 0, 0);
      __builtin_amdgcn_raw_ptr_buffer_load_lds(wR, (LAS void*)(dst + 1024), 16, lp_w, so + 16u * 4096u, 0, 0);
    } else {
      const unsigned so = (unsigned)((u0 + wid) * 8192 + s * 128);
      __builtin_amdgcn_raw_ptr_buffer_load_lds(xR, (LAS void*)dst, 16, lp_x, so, 0, 0);
      __builtin_amdgcn_raw_ptr_buffer_load_lds(xR, (LAS void*)(dst + 1024), 16, lp_x, so + 64u, 0, 0);
    }
  };
  const unsigned craw = ld_sys(ctr);
  unsigned t0 = (unsigned)c, t1 = (unsigned)c + 16u, tprev = 0;
  fetch(t0, 0); fetch(t1, 1);
  const unsigned call = (unsigned)__builtin_amdgcn_readfirstlane(craw) + 1u;
  asm volatile("" :: "s"(call));
  call_out = call;
  __amdgpu_buffer_rsrc_t dstR[7];
  {
    const int rot = (c * 4 + wid) % 7;
#pragma unroll
    for (int q = 0; q < 7; ++q) { const int pr = (rank + 1 + (q + rot) % 7) & 7; dstR[q] = mk_rsrc(p.ws[pr], FLAG_OFF); }
  }
  unsigned flo = 0, fhi = 0;
#pragma unroll
  for (int q = 0; q < 8; ++q) {
    const unsigned long long a = (unsigned long long)(p.ws[q] + FLAG_OFF);
    unsigned lo = (unsigned)a, hi = (unsigned)(a >> 32);
    asm volatile("" : "+s"(lo), "+s"(hi));
    flo = ((lane & 7) == q) ? lo : flo; fhi = ((lane & 7) == q) ? hi : fhi;
  }
  typedef __attribute__((address_space(1))) unsigned short gu16;
  gu16* flp = (gu16*)(((unsigned long long)fhi << 32) | flo);
  const unsigned vo_w = (unsigned)((lane >> 5) * 2048 + ((lane >> 4) & 1) * 256 + (lane & 15) * 16), vo_x = (unsigned)lane * 16u;
  const unsigned dcommon = (call & 1u) * BUF_BYTES;
  const unsigned db_w = dcommon + OPER_BYTES + (unsigned)((wid >> 1) * 1024 + (wid & 1) * 512), db_x = dcommon + (unsigned)wid * 2048u;
  const unsigned rd_w = (unsigned)(size_t)ring + (unsigned)((((lane >> 4) & 1) * 16 + ((lane & 15) >> 1)) * 64 + (lane >> 5) * 16 + (lane & 1) * 8);
  const unsigned rd_x = (unsigned)(size_t)ring + (unsigned)lane * 16u;
  WAIT_L(0); BAR;

  auto signal = [&](unsigned i, unsigned t) {
    unsigned old = 0;
    if (lane == 0) old = __hip_atomic_fetch_add(cnt + (i & 7u), 1u, __ATOMIC_RELAXED, __HIP_MEMORY_SCOPE_WORKGROUP);
    old = __builtin_amdgcn_readfirstlane(old);
    if (old == 3u) {
      int o, eb, s, q4; blk(t, o, eb, s, q4);
      const unsigned idx = (unsigned)((((o * 16 + eb) * 32) + rank * 4 + s) * 4 + q4);
      if (lane < 8 && lane != rank) __hip_atomic_store(flp + idx, (unsigned short)call, __ATOMIC_RELAXED, __HIP_MEMORY_SCOPE_SYSTEM);
      if (lane == 0) { ctl[i & 7u] = 0u; __hip_atomic_fetch_add(cnt + 17, 1u, __ATOMIC_RELAXED, __HIP_MEMORY_SCOPE_WORKGROUP); }
    }
  };
  unsigned i = 0;
#pragma unroll 1
  for (; t0 < 512u; ++i) {
    if (i == 0) WAIT_V(2); else WAIT_V(16);
    int o, eb, s, q4; blk(t0, o, eb, s, q4);
    u32x4 v0, v1;
    if (o) {
      const unsigned a = rd_w + (i & 3u) * 2048u;
      u32x2 a0, a1, b0, b1;
      asm volatile("ds_read_b64_tr_b8 %0, %4\n\tds_read_b64_tr_b8 %1, %4 offset:512\n\t"
                   "ds_read_b64_tr_b8 %2, %4 offset:32\n\tds_read_b64_tr_b8 %3, %4 offset:544\n\ts_waitcnt lgkmcnt(0)"
                   : "=&v"(a0), "=&v"(a1), "=&v"(b0), "=&v"(b1) : "v"(a) : "memory");
      v0 = (u32x4){a0.x, a0.y, a1.x, a1.y}; v1 = (u32x4){b0.x, b0.y, b1.x, b1.y};
    } else {
      const unsigned a = rd_x + (i & 3u) * 2048u;
      asm volatile("ds_read_b128 %0, %2\n\tds_read_b128 %1, %2 offset:1024\n\ts_waitcnt lgkmcnt(0)"
                   : "=&v"(v0), "=&v"(v1) : "v"(a) : "memory");
    }
    const unsigned voff = o ? vo_w : vo_x;
    const unsigned doff = (o ? db_w : db_x) + (unsigned)(((rank * 4 + s) * 256 + eb * 16 + 4 * q4) * 2048);
    const unsigned doff1 = doff + (o ? 4096u : 1024u);
#pragma unroll
    for (int q = 0; q < 7; ++q) {
      __builtin_amdgcn_raw_buffer_store_b128(v0, dstR[q], voff, doff, 17);
      __builtin_amdgcn_raw_buffer_store_b128(v1, dstR[q], voff, doff1, 17);
    }
    if (i >= 1) {
      WAIT_V(16);
      signal(i - 1, tprev);
    }
    unsigned t2 = 512u;
    if (t1 < 512u) {
      while ((unsigned)__builtin_amdgcn_readfirstlane(ctl[16]) <= i + 2u) __builtin_amdgcn_s_sleep(1);
      t2 = (unsigned)__builtin_amdgcn_readfirstlane(ctl[8 + ((i + 2u) & 7u)]);
    }
    fetch(t2, i + 2u);
    tprev = t0; t0 = t1; t1 = t2;
  }
  WAIT_V(0); signal(i - 1, tprev);
}

__device__ __forceinline__ i32x8 ld_frag(LAS unsigned char* a) {
  const i32x4 lo = *(const LAS i32x4*)a, hi = *(const LAS i32x4*)(a + 1024);
  return __builtin_shufflevector(lo, hi, 0, 1, 2, 3, 4, 5, 6, 7);
}

__device__ __forceinline__ i32x8 ld_frag_tr(LAS unsigned char* a) {
  const i32x2 q0 = __builtin_amdgcn_ds_read_tr8_b64_v2i32((LAS i32x2*)a), q1 = __builtin_amdgcn_ds_read_tr8_b64_v2i32((LAS i32x2*)(a + 1024));
  const i32x2 q2 = __builtin_amdgcn_ds_read_tr8_b64_v2i32((LAS i32x2*)(a + 8192)), q3 = __builtin_amdgcn_ds_read_tr8_b64_v2i32((LAS i32x2*)(a + 9216));
  return (i32x8){q0.x, q0.y, q1.x, q1.y, q2.x, q2.y, q3.x, q3.y};
}

__device__ __forceinline__ void gemm_role(const Params& p, LAS unsigned char* lds, unsigned call, int wid, int lane) {
  const int g = blockIdx.x - NCOMM;
  const int L = 30 * (g & 7) + (g >> 3);
  const int tj = (L < 192) ? ((L & 63) >> 2) : (L - 192) / 3, tr = (L < 192) ? ((L >> 6) * 4 + (L & 3)) : 12 + (L - 192) % 3;
  const int wr = wid >> 2, wc = wid & 3;
  const int rank = p.rank & 7;
  char* myws = p.ws[rank];
  const int U = 17 * tr, UX = (tr == 14) ? 255 : U + 8;
  const __amdgpu_buffer_rsrc_t opR = mk_rsrc(myws + (size_t)(call & 1u) * BUF_BYTES, BUF_BYTES);
  const __amdgpu_buffer_rsrc_t xR = mk_rsrc(p.x, SHARD_BYTES), wR = mk_rsrc(p.w, SHARD_BYTES);
  const unsigned voff = (unsigned)lane * 16u;
  volatile LAS unsigned* ctrl = (volatile LAS unsigned*)(lds + L_CTRL);
  unsigned* tmo = (unsigned*)(myws + TMO_OFF);
  const __amdgpu_buffer_rsrc_t tmoR = mk_rsrc(tmo, 64);

  const unsigned sA0a = (unsigned)(U * 2 + wid) * 1024u;
  const unsigned sA0x = (wid < 6) ? (unsigned)(U * 2 + 12 + wid) * 1024u : (unsigned)(UX * 2 + wid - 6) * 1024u;
  const unsigned sA1 = (unsigned)((U + 9) * 2 + wid) * 1024u;
  const unsigned sB = OPER_BYTES + (unsigned)((tj * 16 + 4 * (wid >> 2) + ((wid >> 1) & 1)) * 2048 + (wid & 1) * 1024);
  const unsigned vx = (unsigned)((lane & 15) * 512 + (lane >> 4) * 16);
  const int vch = (lane & 7) ^ (((lane >> 4) & 3) | ((wid & 2) << 1));
  const unsigned vw = (unsigned)((lane >> 3) * 4096 + (vch >> 1) * 64 + (vch & 1) * 16);
  const unsigned lA0a = (unsigned)((U + (wid >> 1)) * 8192 + (wid & 1) * 64);
  const unsigned lA0x = (unsigned)(((wid < 6) ? U + 8 : UX) * 8192 + (wid & 1) * 64);
  const unsigned lA1 = lA0a + 9u * 8192u;
  const unsigned lB = (unsigned)(wid * 8 * 4096 + tj * 256);
  const unsigned lw = (unsigned)wid * 1024u;
#define GLDS(ldsoff, soff) __builtin_amdgcn_raw_ptr_buffer_load_lds(opR, (LAS void*)(lds + (ldsoff)), 16, voff, (soff), 0, 17)
#define GLDX(ldsoff, soff) __builtin_amdgcn_raw_ptr_buffer_load_lds(xR, (LAS void*)(lds + (ldsoff)), 16, vx, (soff), 0, 0)
#define GLDW(ldsoff, soff) __builtin_amdgcn_raw_ptr_buffer_load_lds(wR, (LAS void*)(lds + (ldsoff)), 16, vw, (soff), 0, 0)
#define STAGE_A0(LOC, b, kb) do { if (LOC) { GLDX(L_A0 + (b) * L_A0B + lw, lA0a + (kb) * 128u); GLDX(L_A0 + (b) * L_A0B + lw + 8192, lA0a + 32768u + (kb) * 128u); \
                                             if (wr) GLDX(L_A0 + (b) * L_A0B + lw + 12288, lA0x + (kb) * 128u); } \
                                  else { GLDS(L_A0 + (b) * L_A0B + lw, (kb) + sA0a); GLDS(L_A0 + (b) * L_A0B + lw + 8192, (kb) + sA0a + 8192); \
                                         if (wr) GLDS(L_A0 + (b) * L_A0B + lw + 12288, (kb) + sA0x); } } while (0)
#define STAGE_A1(LOC, b, kb) do { if (LOC) { GLDX(L_A1 + (b) * L_A1B + lw, lA1 + (kb) * 128u); GLDX(L_A1 + (b) * L_A1B + lw + 8192, lA1 + 32768u + (kb) * 128u); } \
                                  else { GLDS(L_A1 + (b) * L_A1B + lw, (kb) + sA1); GLDS(L_A1 + (b) * L_A1B + lw + 8192, (kb) + sA1 + 8192); } } while (0)
#define STAGE_B(LOC, b, bj, kb) do { if (LOC) { GLDW(L_B + (b) * L_BB + (bj) * 16384 + lw, lB + (kb) * 524288u + (bj) * 32); \
                                                GLDW(L_B + (b) * L_BB + (bj) * 16384 + lw + 8192, lB + (kb) * 524288u + (bj) * 32 + 262144u); } \
                                     else { GLDS(L_B + (b) * L_BB + (bj) * 16384 + lw, (kb) + sB + (bj) * 4096); \
                                            GLDS(L_B + (b) * L_BB + (bj) * 16384 + lw + 8192, (kb) + sB + (bj) * 4096 + 16384); } } while (0)
#define WAIT_TILE do { if (wr) WAIT_V(7); else WAIT_V(6); } while (0)

  const bool use5 = (wr == 0) || (tr == 14);
  LAS unsigned char* rA0 = lds + L_A0 + wr * 10240 + lane * 16;
  LAS unsigned char* rA1 = lds + L_A1 + wr * 8192 + lane * 16;
  LAS unsigned char* rB = lds + L_B + wc * 4096 + lane * 16;
  const int fsw = (((lane & 15) >> 2) & 3) | (((lane >> 4) & 1) << 2);
  LAS unsigned char* rBl0 = lds + L_B + ((lane >> 4) * 16 + ((lane & 15) >> 1)) * 128 + (((wc * 2) ^ fsw) * 16) + (lane & 1) * 8;
  LAS unsigned char* rBl1 = lds + L_B + ((lane >> 4) * 16 + ((lane & 15) >> 1)) * 128 + (((wc * 2 + 1) ^ fsw) * 16) + (lane & 1) * 8;
#define LDA0(b) do { _Pragma("unroll") for (int m = 0; m < 4; ++m) At[m] = ld_frag(rA0 + (b) * L_A0B + m * 2048); } while (0)
#define LDA04(b) do { At[4] = ld_frag(rA0 + (b) * L_A0B + 4 * 2048); } while (0)
#define WAIT_HALF(more) do { if (more) { if (wr) WAIT_V(9); else WAIT_V(8); } else WAIT_V(4); } while (0)
#define LDA1(b) do { _Pragma("unroll") for (int m = 0; m < 4; ++m) At[m] = ld_frag(rA1 + (b) * L_A1B + m * 2048); } while (0)
#define LDB(LOC, dst, b, bj) do { if (LOC) { dst[0] = ld_frag_tr(rBl0 + (b) * L_BB + (bj) * 16384); dst[1] = ld_frag_tr(rBl1 + (b) * L_BB + (bj) * 16384); } \
    else { _Pragma("unroll") for (int n = 0; n < 2; ++n) dst[n] = ld_frag(rB + (b) * L_BB + (bj) * 16384 + n * 2048); } } while (0)
#define MMA0(bj, Bt) do { __builtin_amdgcn_s_setprio(1); _Pragma("unroll") for (int m = 0; m < 4; ++m) _Pragma("unroll") for (int n = 0; n < 2; ++n) \
    acc0[bj][m][n] = __builtin_amdgcn_mfma_scale_f32_16x16x128_f8f6f4(Bt[n], At[m], acc0[bj][m][n], 0, 0, 0, 0, 0, 0); \
    if (use5) { _Pragma("unroll") for (int n = 0; n < 2; ++n) \
    acc0[bj][4][n] = __builtin_amdgcn_mfma_scale_f32_16x16x128_f8f6f4(Bt[n], At[4], acc0[bj][4][n], 0, 0, 0, 0, 0, 0); } __builtin_amdgcn_s_setprio(0); } while (0)
#define MMA1(bj, Bt) do { __builtin_amdgcn_s_setprio(1); _Pragma("unroll") for (int m = 0; m < 4; ++m) _Pragma("unroll") for (int n = 0; n < 2; ++n) \
    acc1[bj][m][n] = __builtin_amdgcn_mfma_scale_f32_16x16x128_f8f6f4(Bt[n], At[m], acc1[bj][m][n], 0, 0, 0, 0, 0, 0); __builtin_amdgcn_s_setprio(0); } while (0)

  f32x4 acc0[2][5][2], acc1[2][4][2];
#pragma unroll
  for (int b = 0; b < 2; ++b)
#pragma unroll
    for (int n = 0; n < 2; ++n) {
#pragma unroll
      for (int m = 0; m < 5; ++m) acc0[b][m][n] = (f32x4){0.f, 0.f, 0.f, 0.f};
#pragma unroll
      for (int m = 0; m < 4; ++m) acc1[b][m][n] = (f32x4){0.f, 0.f, 0.f, 0.f};
    }
  i32x8 At[5], B0[2], B1[2];

  const char* fpa = myws + FLAG_OFF + (size_t)(((lane < 32 ? tr : 16 + tj) * 32 + (lane & 31)) * 8);
  const char* fpb = myws + FLAG_OFF + (size_t)(((lane < 32 ? tr + 1 : 16 + tj) * 32 + (lane & 31)) * 8);
  u32x2 pa = {0u, 0u}, pb = {0u, 0u};
  unsigned done = 0xFu << (4 * rank), bad = 0;
  const unsigned short c16 = (unsigned short)call; const u16x2 cc = {c16, c16};
  auto issue_poll = [&]() {
    asm volatile("global_load_dwordx2 %0, %2, off sc0 sc1\n\tglobal_load_dwordx2 %1, %3, off sc0 sc1" : "=&v"(pa), "=&v"(pb) : "v"(fpa), "v"(fpb) : "memory");
  };
  auto pksub = [&](unsigned v) -> unsigned { u16x2 a = __builtin_bit_cast(u16x2, v); a -= cc; return __builtin_bit_cast(unsigned, a); };
  auto ready_mask = [&]() -> unsigned {
    const unsigned w0 = pa[0], w1 = pa[1], w2 = pb[0], w3 = pb[1];
    const unsigned dd = (pksub(w0) | pksub(w1)) | (pksub(w2) | pksub(w3));
    const unsigned long long bl = __builtin_amdgcn_ballot_w64((dd & 0xFFFEFFFEu) == 0u);
    return (unsigned)__builtin_amdgcn_readfirstlane((unsigned)bl & (unsigned)(bl >> 32) & ~done);
  };
  auto decide = [&](int slot) {
    asm volatile("" : "+v"(pa), "+v"(pb));
    unsigned m = ready_mask();
    if (m == 0u) {
      const unsigned long long t0 = __builtin_amdgcn_s_memrealtime();
      while (m == 0u) {
        if (bad == 0u) {
          issue_poll();
          asm volatile("s_waitcnt vmcnt(0)" : "+v"(pa), "+v"(pb) :: "memory");
          m = ready_mask();
          if (m == 0u) {
            const unsigned long long dt = __builtin_amdgcn_s_memrealtime() - t0;
            if (dt > 100000ull) {
              const unsigned tv = (unsigned)__builtin_amdgcn_readfirstlane(__hip_atomic_load(tmo, __ATOMIC_RELAXED, __HIP_MEMORY_SCOPE_AGENT));
              if (tv == call || dt > DEADLINE_TICKS) {
                if (lane == 0) { __hip_atomic_store(tmo, call, __ATOMIC_RELAXED, __HIP_MEMORY_SCOPE_AGENT); ctrl[2] = 1u; }
                bad = 1u;
              }
              __builtin_amdgcn_s_sleep(64);
            }
            __builtin_amdgcn_s_sleep(2);
          }
        } else {
          m = ~done;
        }
        m = (unsigned)__builtin_amdgcn_readfirstlane(m);
        bad = (unsigned)__builtin_amdgcn_readfirstlane(bad);
      }
    }
    const unsigned ks = (unsigned)__builtin_ctz(m);
    done = (unsigned)__builtin_amdgcn_readfirstlane(done | (1u << ks));
    if (lane == 0) ctrl[slot] = ks;
    issue_poll();
  };
#define READ_D(slot) ((unsigned)__builtin_amdgcn_readfirstlane(ctrl[slot]) * (512u * 1024u))

  const float scale = __builtin_bit_cast(float, __builtin_amdgcn_readfirstlane(__builtin_bit_cast(int, p.sx[0] * p.sw[0])));
  if (wid == 0) { if (lane == 0) ctrl[2] = 0u; issue_poll(); }
  STAGE_B(1, 0, 0, 0u); STAGE_A0(1, 0, 0u); STAGE_B(1, 0, 1, 0u); STAGE_A1(1, 0, 0u);
  if (wr == 1) BAR;
  WAIT_V(4); BAR;
  STAGE_B(1, 1, 0, 1u); STAGE_A0(1, 1, 1u); STAGE_B(1, 1, 1, 1u);
  WAIT_TILE; BAR;

  LDB(1, B0, 0, 0); LDA04(0);
#define TRIP(RL, S1L, SNL, more) do { \
    unsigned kb2 = 2u, kb3 = 3u; \
      \
    if (!(SNL) && wid == 0 && (more)) decide(0); \
    if (wid == 0 && !(more) && lane == 0) __builtin_amdgcn_raw_ptr_buffer_load_lds(tmoR, (LAS void*)(lds + L_CTRL + 12), 4, 0u, 0u, 0, 16);     \
    LDA0(0); STAGE_A1(S1L, 1, kb1); \
    BAR; WAIT_L(0); MMA0(0, B0); BAR; SCHED; \
      \
    if (!(SNL) && (more)) kb2 = READ_D(0); \
    LDB(RL, B1, 0, 1); if (more) STAGE_B(SNL, 0, 0, kb2); \
    BAR; WAIT_L(0); MMA0(1, B1); BAR; \
      \
    LDA1(0); if (more) STAGE_A0(SNL, 0, kb2); \
    WAIT_HALF(more); BAR; WAIT_L(0); MMA1(0, B0); BAR; SCHED; \
      \
    LDB(RL, B0, 1, 0); LDA04(1); \
    if (more) { STAGE_B(SNL, 0, 1, kb2); WAIT_TILE; } else { WAIT_V(0); } \
    BAR; MMA1(1, B1); BAR; \
      \
    if (!(SNL) && wid == 0 && (more)) decide(1); \
    LDA0(1); if (more) STAGE_A1(SNL, 0, kb2); \
    BAR; WAIT_L(0); MMA0(0, B0); BAR; SCHED; \
      \
    if (!(SNL) && (more)) kb3 = READ_D(1); \
    LDB(RL, B1, 1, 1); if (more) STAGE_B(SNL, 1, 0, kb3); \
    BAR; WAIT_L(0); MMA0(1, B1); BAR; \
      \
    LDA1(1); if (more) { STAGE_A0(SNL, 1, kb3); WAIT_HALF(true); } \
    BAR; WAIT_L(0); MMA1(0, B0); BAR; SCHED; \
      \
    if (more) { LDB(SNL, B0, 0, 0); LDA04(0); STAGE_B(SNL, 1, 1, kb3); WAIT_TILE; } \
    BAR; MMA1(1, B1); BAR; \
    kb1 = kb3; } while (0)
  unsigned kb1 = 1u;
#pragma unroll 1
  for (int gi = 0; gi < 2; ++gi) { const bool snl = gi == 0; TRIP(1, 1, snl, true); }
#pragma unroll 1
  for (int it = 2; it < 15; ++it) TRIP(0, 0, 0, true);
  { int once = 1; asm volatile("" : "+s"(once));
#pragma unroll 1
    for (int z = 0; z < once; ++z) TRIP(0, 0, 0, false); }
  if (wr == 0) BAR;

  const float escale = (ctrl[2] != 0u || ctrl[3] == call) ? __builtin_nanf("") : scale;
  int el = lane; asm volatile("" : "+v"(el));
  const int fr = el & 15, fq = el >> 4;
  LAS unsigned char* ewr = lds + wid * 8704 + fr * 272 + fq * 16;
  LAS unsigned char* erd = lds + wid * 8704 + fq * 272 + fr * 16;
  float* obase = p.out + (size_t)fq * 4096 + tj * 256 + wc * 64 + fr * 4;
  auto pass = [&](int j, f32x4& a00, f32x4& a01, f32x4& a10, f32x4& a11, int unit, bool live) {
    if (!live) return;
    const f32x4* src[4] = {&a00, &a01, &a10, &a11};
#pragma unroll
    for (int q = 0; q < 4; ++q) {
      f32x4 y = *src[q] * escale;
#pragma unroll
      for (int e = 0; e < 4; ++e) y[e] = (y[e] < 0.f) ? 0.f : y[e];
      *(LAS f32x4*)(ewr + (j & 1) * 4352 + q * 64) = y;
    }
    f32x4 r[4];
#pragma unroll
    for (int i = 0; i < 4; ++i) r[i] = *(const LAS f32x4*)(erd + (j & 1) * 4352 + i * 4 * 272);
#pragma unroll
    for (int i = 0; i < 4; ++i) asm volatile("global_store_dwordx4 %0, %1, off sc1\n\ts_nop 1" :: "v"(obase + (size_t)(unit * 16 + i * 4) * 4096), "v"(r[i]) : "memory");
  };
#pragma unroll
  for (int m = 0; m < 5; ++m) {
    const int slot = wr * 5 + m;
    pass(m, acc0[0][m][0], acc0[0][m][1], acc0[1][m][0], acc0[1][m][1], (slot < 9) ? U + slot : 255, slot < 9 || tr == 14);
  }
#pragma unroll
  for (int m = 0; m < 4; ++m) pass(5 + m, acc1[0][m][0], acc1[0][m][1], acc1[1][m][0], acc1[1][m][1], U + 9 + wr * 4 + m, true);
}
}

__global__ void __launch_bounds__(512, 2) fused_gemm_ag_kernel(Params p) {
  extern __shared__ __attribute__((aligned(16))) unsigned char smem_dyn[];
  LAS unsigned char* lds = (LAS unsigned char*)smem_dyn;
  const int wid = __builtin_amdgcn_readfirstlane(threadIdx.x >> 6), lane = threadIdx.x & 63;
  unsigned* ctr = (unsigned*)(p.ws[p.rank & 7] + CTR_OFF) + blockIdx.x * 16;
  unsigned call;
  if (blockIdx.x < NCOMM) comm_role(p, lds, ctr, call, wid, lane);
  else { call = (unsigned)__builtin_amdgcn_readfirstlane(ld_sys(ctr)) + 1u;
    gemm_role(p, lds, call, wid, lane);
  }
  if (threadIdx.x == 0) __hip_atomic_store(ctr, call, __ATOMIC_RELAXED, __HIP_MEMORY_SCOPE_SYSTEM);
}

extern "C" void kernel_launch(int rank, int nranks, ncclComm_t comm,
                              hipStream_t stream,
                              void* const* d_in, const int* in_sizes, int n_in,
                              void* d_out, int out_size,
                              const int* in_dtypes, int out_dtype,
                              void* const* d_out_all,
                              void* const* d_ws_all, size_t ws_bytes) {
  (void)comm; (void)in_sizes; (void)n_in; (void)out_size; (void)in_dtypes; (void)out_dtype; (void)d_out_all;
  if (nranks != NRANK || (unsigned)rank >= (unsigned)NRANK || ws_bytes < (size_t)WS_USED) {
    fprintf(stderr, "unsupported configuration\n"); return;
  }
  HIP_CHECK(hipFuncSetAttribute((const void*)fused_gemm_ag_kernel, hipFuncAttributeMaxDynamicSharedMemorySize, (int)LDS_BYTES));
  Params p;
  p.x = (const unsigned char*)d_in[0]; p.w = (const unsigned char*)d_in[1];
  p.sx = (const float*)d_in[2]; p.sw = (const float*)d_in[3];
  p.out = (float*)d_out; p.rank = rank;
  for (int i = 0; i < NRANK; ++i) p.ws[i] = (char*)d_ws_all[i];
  hipLaunchKernelGGL(fused_gemm_ag_kernel, dim3(NWG), dim3(512), LDS_BYTES, stream, p);
  HIP_CHECK(hipGetLastError());
}
```
